```python
import numpy as np

kernel_name = "gpu_a2a_gemm_m4096_k4096_n4096_f32_reluqe4m3"


ABSMAX_RTOL = 0.09

COMPUTE_DTYPE = "float32"
EPILOGUE = "relu_quant_e4m3"
IN_QUANT = None
OUT_QUANT = "e4m3"
IN_DTYPES = {"x": "float32", "w_mat": "float32"}
WORLD_SIZE = 8

def setup_inputs():

    x = np.random.randn(4096, 4096).astype(np.float32)
    w_mat = (np.random.randn(4096, 4096) / np.sqrt(4096)).astype(np.float32)
    return {"x": x, "w_mat": w_mat}

def _snap_fp8(x, mant_bits, max_abs, min_normal):
    a = np.abs(x.astype(np.float64))
    _m, _e = np.frexp(a)
    q = np.where(a >= min_normal, np.ldexp(1.0, _e - 1 - mant_bits),
                 min_normal * 2.0 ** -mant_bits)
    snapped = np.minimum(np.round(a / q) * q, max_abs)
    return (np.sign(x) * snapped).astype(np.float32)
def _quant_dequant(y, amax):
    scale = amax / 448.0
    q = _snap_fp8(y / scale, 3, 448.0, 2.0 ** -6)
    return (q * scale).astype(np.float32)
def _epilogue(y):
    y = np.maximum(y, 0.0)
    return _quant_dequant(y, np.abs(y).max())

def reference(x, w_mat):
    y = (x @ w_mat).astype(np.float32)
    return _epilogue(y)
```

```cpp
#include <hip/hip_runtime.h>
#include <rccl/rccl.h>
#include <cstdio>
#include <cstdint>

typedef float    f32x4 __attribute__((ext_vector_type(4)));
typedef float    f32x2 __attribute__((ext_vector_type(2)));
typedef unsigned u32x4 __attribute__((ext_vector_type(4)));
typedef unsigned u32x2 __attribute__((ext_vector_type(2)));
typedef __bf16   bf16x8 __attribute__((ext_vector_type(8)));
typedef __bf16   bf16x2 __attribute__((ext_vector_type(2)));

namespace {

constexpr int NRK   = 8;
constexpr int NWG   = 256;
constexpr int NST   = 16;
constexpr int NU    = 12;
constexpr int NUN   = 2 * NST;

constexpr unsigned OFF_X   = 0;
constexpr unsigned OFF_XF  = 4u << 20;
constexpr unsigned OFF_AM  = OFF_XF + 16384u;
constexpr unsigned OFF_CNT = OFF_AM + 16384u;
constexpr unsigned OFF_RF  = OFF_CNT + 32768u;
constexpr unsigned OFF_TMO = OFF_RF + 65536u;
constexpr unsigned OFF_RED = 8u << 20;
constexpr unsigned WS_USED = OFF_RED + (16u << 20);

#define ABL 0
#define XAUX 17
#define XSC "sc0 sc1"
#define XREC WS_USED
#define WREC (4096u * 4096u * 4u)
#define STAMP(i) do {} while (0)
#define VSTAMP(i, v) do {} while (0)
#define CLK() 0ull
constexpr int AUX_SYS = 17;
constexpr int AUX_AGT = 16;
constexpr unsigned long long DEADLINE_TICKS = 50000000ull;
constexpr unsigned OOB = 0x40000000u;

struct Params {
    const float* x;
    const float* w;
    unsigned char* out;
    char* myws;
    char* ws[NRK];
    int rank;
};

typedef __amdgpu_buffer_rsrc_t rsrc_t;

__device__ __forceinline__ rsrc_t mkrsrc(const void* p, unsigned bytes) {
    return __builtin_amdgcn_make_buffer_rsrc(const_cast<void*>(p), (short)0, (int)bytes, 0x00020000);
}
__device__ __forceinline__ unsigned pk2(float a, float b) {
    bf16x2 r = __builtin_convertvector((f32x2){a, b}, bf16x2);
    return __builtin_bit_cast(unsigned, r);
}
__device__ __forceinline__ float wave_max(float v) {
#pragma unroll
    for (int o = 32; o >= 1; o >>= 1) v = fmaxf(v, __shfl_xor(v, o, 64));
    return v;
}
template <int N> __device__ __forceinline__ void wait_vm() {
    asm volatile("s_waitcnt vmcnt(%0)" :: "i"(N) : "memory");
}
__device__ __forceinline__ void wait_lgkm0() { asm volatile("s_waitcnt lgkmcnt(0)" ::: "memory"); }

__device__ __forceinline__ int lds_ld(int* p) {
    return __builtin_amdgcn_readfirstlane(__hip_atomic_load(p, __ATOMIC_RELAXED, __HIP_MEMORY_SCOPE_WORKGROUP));
}
__device__ __forceinline__ void lds_wait_ge(int* p, int v) {
    while (lds_ld(p) < v) __builtin_amdgcn_s_sleep(1);
    asm volatile("" ::: "memory");
}
__device__ __forceinline__ void lds_add1(int* p, int lane) {
    if (lane == 0) __hip_atomic_fetch_add(p, 1, __ATOMIC_RELAXED, __HIP_MEMORY_SCOPE_WORKGROUP);
}

typedef int i32x4 __attribute__((ext_vector_type(4)));
__device__ __forceinline__ void lds_set(int* p, int v) {
    __hip_atomic_store(p, v, __ATOMIC_RELAXED, __HIP_MEMORY_SCOPE_WORKGROUP);
}
__device__ __forceinline__ i32x4 lds_ld4(const int* p) {
    typedef const volatile __attribute__((address_space(3))) i32x4* lp_t;
    return *(lp_t)(p);
}
__device__ __forceinline__ int units_ready(i32x4 v) {
    return __builtin_amdgcn_readfirstlane(min(min(v[0], v[1]), v[2]));
}
__device__ __forceinline__ int cons_done(const int* p) {
    const i32x4 v = lds_ld4(p);
    return __builtin_amdgcn_readfirstlane(min(min(v[0], v[1]), min(v[2], v[3])));
}

__device__ __forceinline__ int stage_of(int c, int r) {
    const int g = c & 3, j = c >> 2;
    if (r == 0) return j;
    const int t = (c - r) & 7;
    return 2 + 2 * t + j - (((g < t) || (g == t && j == 1)) ? 1 : 0) - ((g + 4 < t) ? 1 : 0);
}

#define LAGV 1
constexpr int PUSH_LAG = LAGV;
__host__ __device__ constexpr int push_cnt(int j) {
    return PUSH_LAG + ((j - PUSH_LAG) < PUSH_LAG ? (j - PUSH_LAG) : PUSH_LAG);
}

struct PushCtx {
    rsrc_t rdst; unsigned vo_data; unsigned so[8]; unsigned fo[8]; unsigned seq; int lane;
};
template <int J> struct PushRounds {
    __device__ __forceinline__ static void run(const f32x4 (&xv)[32], const PushCtx& C) {
        if constexpr (J < 16) {
            u32x4 d;
            d[0] = pk2(xv[2 * J][0], xv[2 * J][1]);
            d[1] = pk2(xv[2 * J][2], xv[2 * J][3]);
            d[2] = pk2(xv[2 * J + 1][0], xv[2 * J + 1][1]);
            d[3] = pk2(xv[2 * J + 1][2], xv[2 * J + 1][3]);
            __builtin_amdgcn_raw_buffer_store_b128(d, C.rdst, C.vo_data, C.so[J >> 1] + (J & 1) * 16384u, AUX_SYS);
        }
        if constexpr (J >= PUSH_LAG) {
            if constexpr (J < 16) wait_vm<push_cnt(J)>();
            else wait_vm<2 * PUSH_LAG - 1 - (J - 16)>();
            constexpr int F = J - PUSH_LAG;
            if (C.lane == 0)
                __builtin_amdgcn_raw_buffer_store_b32(C.seq, C.rdst, 0, C.fo[F >> 1] + (F & 1) * 64u, AUX_SYS);
        }
        if constexpr (J + 1 < 16 + PUSH_LAG) PushRounds<J + 1>::run(xv, C);
    }
};

struct Smem {
    u32x4 ring[NU][512];
    f32x4 own[32 * 64];
    int prog[8];
    int xdone;
    int tab[NST];
    int avail, fail;
    float wmax[8], gmax[8];
};

__global__ void __launch_bounds__(512, 2) a2a_gemm_kernel(Params P) {
    __shared__ __attribute__((aligned(16))) Smem sm;

    const int tid  = threadIdx.x;
    const int lane = tid & 63;
    const int wave = __builtin_amdgcn_readfirstlane(tid >> 6);
    const int b    = blockIdx.x;
    const int rank = P.rank;

    const rsrc_t rws = mkrsrc(P.myws, WS_USED);

    const int jb    = b >> 3;
    const int g     = (b & 7) >> 1;
    const int ntile = (b & 1) * 16 + (jb >> 1);
    const int mh    = jb & 1;
    const int tile  = ntile * 2 + mh;

    const int d      = b & 7;
    const int rowblk = b >> 3;
    const int pmh = rowblk >> 4, pwq = (rowblk >> 2) & 3, pmf = rowblk & 3;
    const int pr_ = (rank - d) & 7;
    f32x4 xv[32];
    if (wave == 6) {
        const rsrc_t rx = mkrsrc(P.x, 4096u * 512u * 4u);
        const unsigned vox = (unsigned)(512 * d + 16 * rowblk + (lane & 15)) * 2048u + (unsigned)(lane >> 4) * 32u;
#pragma unroll
        for (int t = 0; t < 8; ++t) {
            const unsigned xo = (unsigned)((t + pr_) & 7) * 256u;
            xv[4 * t]     = __builtin_bit_cast(f32x4, __builtin_amdgcn_raw_buffer_load_b128(rx, vox, xo, 0));
            xv[4 * t + 1] = __builtin_bit_cast(f32x4, __builtin_amdgcn_raw_buffer_load_b128(rx, vox, xo + 16u, 0));
            xv[4 * t + 2] = __builtin_bit_cast(f32x4, __builtin_amdgcn_raw_buffer_load_b128(rx, vox, xo + 128u, 0));
            xv[4 * t + 3] = __builtin_bit_cast(f32x4, __builtin_amdgcn_raw_buffer_load_b128(rx, vox, xo + 144u, 0));
        }
    }
    unsigned seq = __builtin_amdgcn_raw_buffer_load_b32(rws, 0, OFF_CNT + b * 128u, AUX_AGT);
    if (tid < 8) sm.prog[tid] = 0;
    if (tid == 32) { sm.avail = 0; sm.fail = 0; sm.xdone = 0; }
    if (tid >= 64 && tid < 80) {
        const int i = tid - 64, t = i >> 1, j = i & 1;
        const int r = (g + 4 * j - t) & 7;
        sm.tab[stage_of(g + 4 * j, r)] = 512 * ((rank + r) & 7) + 64 * (g + 4 * j);
    }
    seq = __builtin_amdgcn_readfirstlane(seq) + 1u;
    __syncthreads();

    f32x4 acc[4][8];
#pragma unroll
    for (int f = 0; f < 4; ++f)
#pragma unroll
        for (int i = 0; i < 8; ++i) acc[f][i] = (f32x4){0.f, 0.f, 0.f, 0.f};

    if (wave < 4) {
#define CPRIO 0
#define HPRIO 2
        __builtin_amdgcn_s_setprio(CPRIO);
        const unsigned vox = (unsigned)lane * 16u;
        const unsigned xbase = OFF_X + (unsigned)((mh * 4 + g) * 16) * 32768u + (unsigned)wave * 4096u;
        u32x4 a0[4], a1[4], a2[4], a3[4];
        bf16x8 fb[8];
        unsigned long long cw = 0;
        int rdw = 0, rdx = 0;
        auto need = [&](int nw, int nx_) __attribute__((always_inline)) {
            if (rdw < nw || rdx < nx_) {
                unsigned long long t_ = CLK();
                for (;;) {
                    const i32x4 v = lds_ld4(&sm.prog[4]);
                    rdw = __builtin_amdgcn_readfirstlane(min(v[0], v[1]));
                    rdx = __builtin_amdgcn_readfirstlane(v[2]);
                    if (rdw >= nw && rdx >= nx_) break;
                    __builtin_amdgcn_s_sleep(1);
                }
                cw += CLK() - t_;
            }
            asm volatile("" ::: "memory");
        };
        auto issueA = [&](u32x4 (&av_)[4], int n) __attribute__((always_inline)) {
            const unsigned sx = n < NUN ? xbase + (unsigned)n * 16384u : OOB;
#pragma unroll
            for (int f = 0; f < 4; ++f)
                av_[f] = __builtin_amdgcn_raw_buffer_load_b128(rws, vox + f * 1024u, sx, XAUX);
        };
        auto block = [&](const u32x4 (&ca)[4], u32x4 (&na)[4], int u, int nslot) __attribute__((always_inline)) {
            need(u + 2 < NUN ? u + 2 : NUN, u + 4 < NUN ? u + 4 : NUN);
            __builtin_amdgcn_sched_barrier(0);
            const unsigned sx = u + 3 < NUN ? xbase + (unsigned)(u + 3) * 16384u : OOB;
            const u32x4* rb = &sm.ring[nslot][lane];
            i32x4 fl_v;
#pragma unroll
            for (int i = 0; i < 8; ++i) {
#pragma unroll
                for (int f = 0; f < 4; ++f)
                    asm volatile("v_mfma_f32_16x16x32_bf16 %0, %1, %2, %0" : "+a"(acc[f][i]) : "v"(ca[f]), "v"(fb[i]));
                __builtin_amdgcn_sched_barrier(0);
                fb[i] = __builtin_bit_cast(bf16x8, rb[i * 64]);
                if (i < 4) na[i] = __builtin_amdgcn_raw_buffer_load_b128(rws, vox + i * 1024u, sx, XAUX);
                if (i == 5) fl_v = lds_ld4(&sm.prog[4]);
                __builtin_amdgcn_sched_barrier(0);
            }
            __builtin_amdgcn_sched_barrier(0);
            return fl_v;
        };
        auto after = [&](i32x4 fl_v, int u) __attribute__((always_inline)) {
            lds_set(&sm.prog[wave], u + 1);
            rdw = __builtin_amdgcn_readfirstlane(min(fl_v[0], fl_v[1]));
            rdx = __builtin_amdgcn_readfirstlane(fl_v[2]);
        };
        need(0, 1); issueA(a0, 0);
        need(0, 2); issueA(a1, 1);
        need(0, 3); issueA(a2, 2);
        need(1, 0);
        if (wave == 0) STAMP(2);
#pragma unroll
        for (int i = 0; i < 8; ++i) fb[i] = __builtin_bit_cast(bf16x8, sm.ring[0][i * 64 + lane]);
        unsigned long long c0 = CLK(), r0 = __builtin_amdgcn_s_memrealtime();
        int slot = 0;
        for (int u = 0; u < NUN; u += 4) {
            i32x4 fv;
            fv = block(a0, a3, u, slot + 1);     after(fv, u);
            fv = block(a1, a0, u + 1, slot + 2); after(fv, u + 1);
            fv = block(a2, a1, u + 2, slot + 3); after(fv, u + 2);
            slot += 4; if (slot == NU) slot = 0;
            fv = block(a3, a2, u + 3, slot);     after(fv, u + 3);
        }
        asm volatile("s_nop 15\n\ts_nop 15"
                     : "+a"(acc[0][6]), "+a"(acc[1][6]), "+a"(acc[2][6]), "+a"(acc[3][6]),
                       "+a"(acc[0][7]), "+a"(acc[1][7]), "+a"(acc[2][7]), "+a"(acc[3][7]) :: "memory");
        if (wave == 0) { VSTAMP(0, CLK() - c0); VSTAMP(1, cw); VSTAMP(2, __builtin_amdgcn_s_memrealtime() - r0); }
        __builtin_amdgcn_s_setprio(0);
        if (wave == 0) STAMP(3);
        if (wave == g) {
            f32x4* own = &sm.own[0];
#pragma unroll
            for (int f = 0; f < 4; ++f)
#pragma unroll
                for (int i = 0; i < 8; ++i) own[(f * 8 + i) * 64 + lane] = acc[f][i];
        } else {
            const unsigned so = OFF_RED + (unsigned)((tile * 4 + wave) * 4 + g) * 16384u;
#pragma unroll
            for (int f = 0; f < 4; ++f)
#pragma unroll
                for (int ip = 0; ip < 4; ++ip) {
                    u32x4 t;
                    t[0] = pk2(acc[f][2 * ip][0], acc[f][2 * ip][1]);
                    t[1] = pk2(acc[f][2 * ip][2], acc[f][2 * ip][3]);
                    t[2] = pk2(acc[f][2 * ip + 1][0], acc[f][2 * ip + 1][1]);
                    t[3] = pk2(acc[f][2 * ip + 1][2], acc[f][2 * ip + 1][3]);
                    __builtin_amdgcn_raw_buffer_store_b128(t, rws, vox + ip * 1024u, so + f * 4096u, AUX_AGT);
                }
            wait_vm<0>();
            if (lane == 0)
                __builtin_amdgcn_raw_buffer_store_b32(seq, rws, 0, OFF_RF + (unsigned)((tile * 4 + wave) * 4 + g) * 64u, AUX_AGT);
        }
    } else if (wave < 6) {
        __builtin_amdgcn_s_setprio(HPRIO);
        const int nh = wave - 4;
        const rsrc_t rw = mkrsrc(P.w, WREC);
        const unsigned vow = (unsigned)(lane >> 4) * (8u * 16384u) + (unsigned)(lane & 15) * 16u;
        const unsigned wcol = (unsigned)(ntile * 128 + nh * 64) * 4u;
        constexpr int DW = 4;
        unsigned long long lw = 0, l0 = CLK();
        f32x4 wb[DW][8];
        const int tabv = sm.tab[lane & 15];
        auto issueW = [&](f32x4 (&wv)[8], int u) __attribute__((always_inline)) {
            unsigned sw = OOB;
            if (u < NUN) sw = (unsigned)(__builtin_amdgcn_readlane(tabv, u >> 1) + 32 * (u & 1)) * 16384u + wcol;
#pragma unroll
            for (int j = 0; j < 8; ++j)
                wv[j] = __builtin_bit_cast(f32x4, __builtin_amdgcn_raw_buffer_load_b128(rw, vow, sw + (unsigned)j * 16384u, 0));
        };
        int cdone = 0;
        auto fill = [&](const f32x4 (&wv)[8], int slot, int u) __attribute__((always_inline)) {
            if (cdone < u - (NU - 1)) {
                unsigned long long t_ = CLK();
                while ((cdone = cons_done(&sm.prog[0])) < u - (NU - 1)) __builtin_amdgcn_s_sleep(1);
                lw += CLK() - t_;
            }
            asm volatile("" ::: "memory");
#pragma unroll
            for (int i = 0; i < 4; ++i) {
                u32x4 t;
                t[0] = pk2(wv[0][i], wv[1][i]);
                t[1] = pk2(wv[2][i], wv[3][i]);
                t[2] = pk2(wv[4][i], wv[5][i]);
                t[3] = pk2(wv[6][i], wv[7][i]);
                sm.ring[slot][(nh * 4 + i) * 64 + lane] = t;
            }
            asm volatile("" ::: "memory");
            lds_set(&sm.prog[4 + nh], u + 1);
            if (nh == 0 && u == 1) STAMP(5);
            if (nh == 0 && u == 5) STAMP(6);
        };
#define GATE 6
        int u0 = 0, fslot = 0;
        for (;;) {
            issueW(wb[0], u0);
            fill(wb[0], fslot, u0);
            ++u0; if (++fslot == NU) fslot = 0;
            if (u0 >= NU - 2 || __builtin_amdgcn_readfirstlane(lds_ld4(&sm.prog[4])[2]) >= GATE) break;
        }
#pragma unroll
        for (int dd = 0; dd < 4; ++dd) issueW(wb[dd], u0 + dd);
        for (int u = u0; u < NUN; u += 4) {
#pragma unroll
            for (int dd = 0; dd < 4; ++dd) {
                if (u + dd < NUN) fill(wb[dd], fslot, u + dd);
                if (++fslot == NU) fslot = 0;
                issueW(wb[dd], u + dd + 4);
            }
        }
        if (nh == 0) { VSTAMP(3, CLK() - l0); VSTAMP(4, lw); }
        STAMP(11 + nh);
    } else if (wave == 6) {
        __builtin_amdgcn_s_setprio(HPRIO);
        PushCtx C;
#pragma unroll
        for (int t = 0; t < 8; ++t) {
            const int c = (t + pr_) & 7;
            const unsigned blk = (unsigned)((pmh * 4 + (c & 3)) * 16 + stage_of(c, pr_));
            C.so[t] = OFF_X + blk * 32768u + (unsigned)pwq * 4096u + (unsigned)pmf * 1024u;
            C.fo[t] = OFF_XF + (blk * 32u + (unsigned)pwq * 4u + (unsigned)pmf) * 4u;
        }
        char* dst = P.myws;
#pragma unroll
        for (int q = 0; q < NRK; ++q) if (q == d) dst = P.ws[q];
        C.rdst = mkrsrc(dst, WS_USED);
        C.vo_data = (unsigned)lane * 16u;
        C.seq = seq; C.lane = lane;
        if (pr_ == 0) {
#pragma unroll
            for (int J = 0; J < 16; ++J) {
                u32x4 dd;
                dd[0] = pk2(xv[2 * J][0], xv[2 * J][1]);
                dd[1] = pk2(xv[2 * J][2], xv[2 * J][3]);
                dd[2] = pk2(xv[2 * J + 1][0], xv[2 * J + 1][1]);
                dd[3] = pk2(xv[2 * J + 1][2], xv[2 * J + 1][3]);
                __builtin_amdgcn_raw_buffer_store_b128(dd, C.rdst, C.vo_data, C.so[J >> 1] + (J & 1) * 16384u, AUX_SYS);
            }
            wait_vm<0>();
#pragma unroll
            for (int J = 0; J < 16; ++J)
                if (lane == 0)
                    __builtin_amdgcn_raw_buffer_store_b32(C.seq, C.rdst, 0, C.fo[J >> 1] + (J & 1) * 64u, AUX_SYS);
        } else
        PushRounds<0>::run(xv, C);
        STAMP(1);
    } else {
        __builtin_amdgcn_s_setprio(HPRIO);
        const unsigned long long t0 = __builtin_amdgcn_s_memrealtime();
        const unsigned fo = OFF_XF + (unsigned)(mh * 4 + g) * 2048u;
        int av = 0;
        while (av < NUN) {
            const unsigned vf = (lane < 16 && av + (lane >> 1) < NUN) ? (unsigned)lane * 32u : OOB;
            const unsigned sf = fo + (unsigned)av * 64u;
            u32x4 f0 = __builtin_amdgcn_raw_buffer_load_b128(rws, vf, sf, AUX_SYS);
            u32x4 f1 = __builtin_amdgcn_raw_buffer_load_b128(rws, vf + 16u, sf, AUX_SYS);
            const bool ok = (f0[0] == seq) & (f0[1] == seq) & (f0[2] == seq) & (f0[3] == seq) &
                            (f1[0] == seq) & (f1[1] == seq) & (f1[2] == seq) & (f1[3] == seq);
            const unsigned m = (unsigned)__ballot(ok) & 0xffffu;
            const unsigned m2 = m & (m >> 1);
            const unsigned xm = (~m2 & 0x5555u) | 0x10000u;
            const int na = __builtin_amdgcn_readfirstlane(av + (__builtin_ctz(xm) >> 1));
            if (na > av) {
                if (av < 4 && na >= 4) STAMP(13);
                if (av < 8 && na >= 8) STAMP(14);
                if (na >= NUN) STAMP(15);
                av = na;
                lds_set(&sm.prog[6], av);
            } else if (__builtin_amdgcn_s_memrealtime() - t0 <= 100000ull) {
                __builtin_amdgcn_s_sleep(2);
            } else {
                const unsigned tm = __builtin_amdgcn_readfirstlane(
                    __builtin_amdgcn_raw_buffer_load_b32(rws, 0, OFF_TMO, AUX_AGT));
                if (tm == seq || __builtin_amdgcn_s_memrealtime() - t0 > DEADLINE_TICKS) {
                    if (lane == 0) {
                        __builtin_amdgcn_raw_buffer_store_b32(seq, rws, 0, OFF_TMO, AUX_AGT);
                        __hip_atomic_store(&sm.fail, 1, __ATOMIC_RELAXED, __HIP_MEMORY_SCOPE_WORKGROUP);
                    }
                    av = NUN;
                    lds_set(&sm.prog[6], av);
                }
            }
            av = __builtin_amdgcn_readfirstlane(av);
        }
        STAMP(10);
    }
    __builtin_amdgcn_s_setprio(0);
    __syncthreads();

    const int fb = (wave >> 1) * 8 + (wave & 1) * 4;
    f32x4 v[4];
    {
        const unsigned long long t0 = __builtin_amdgcn_s_memrealtime();
        for (;;) {
            const unsigned f = __builtin_amdgcn_raw_buffer_load_b32(rws, (unsigned)(lane & 3) * 64u,
                                                                    OFF_RF + (unsigned)(tile * 4 + g) * 256u, AUX_AGT);
            if (__all((f == seq) | ((lane & 3) == g))) break;
            if (__builtin_amdgcn_s_memrealtime() - t0 < 100000ull) continue;
            const unsigned tm = __builtin_amdgcn_readfirstlane(
                __builtin_amdgcn_raw_buffer_load_b32(rws, 0, OFF_TMO, AUX_AGT));
            if (tm == seq || __builtin_amdgcn_s_memrealtime() - t0 > DEADLINE_TICKS) {
                if (lane == 0) {
                    __builtin_amdgcn_raw_buffer_store_b32(seq, rws, 0, OFF_TMO, AUX_AGT);
                    __hip_atomic_store(&sm.fail, 1, __ATOMIC_RELAXED, __HIP_MEMORY_SCOPE_WORKGROUP);
                }
                break;
            }
            __builtin_amdgcn_s_sleep(1);
        }
        asm volatile("" ::: "memory");
        u32x4 pr[3][2];
#pragma unroll
        for (int s = 0; s < 3; ++s) {
            const int src = s + (s >= g ? 1 : 0);
            const unsigned so = OFF_RED + (unsigned)((tile * 4 + g) * 4 + src) * 16384u + (unsigned)wave * 2048u;
#pragma unroll
            for (int i = 0; i < 2; ++i)
                pr[s][i] = __builtin_amdgcn_raw_buffer_load_b128(rws, (unsigned)lane * 16u + i * 1024u, so, AUX_AGT);
        }
        const f32x4* own = &sm.own[0];
#pragma unroll
        for (int i = 0; i < 4; ++i) v[i] = own[(fb + i) * 64 + lane];
#pragma unroll
        for (int s = 0; s < 3; ++s)
#pragma unroll
            for (int i = 0; i < 4; ++i) {
                const unsigned lo = pr[s][i >> 1][(i & 1) * 2], hi = pr[s][i >> 1][(i & 1) * 2 + 1];
                v[i][0] += __builtin_bit_cast(float, lo << 16);
                v[i][1] += __builtin_bit_cast(float, lo & 0xffff0000u);
                v[i][2] += __builtin_bit_cast(float, hi << 16);
                v[i][3] += __builtin_bit_cast(float, hi & 0xffff0000u);
            }
    }
    float vmax = 0.f;
#pragma unroll
    for (int i = 0; i < 4; ++i)
#pragma unroll
        for (int r = 0; r < 4; ++r) { v[i][r] = fmaxf(v[i][r], 0.f); vmax = fmaxf(vmax, v[i][r]); }
    vmax = wave_max(vmax);
    if (lane == 0) sm.wmax[wave] = vmax;
    __syncthreads();
    if (wave == 0) STAMP(7);

    {
        float m = sm.wmax[0];
#pragma unroll
        for (int q = 1; q < 8; ++q) m = fmaxf(m, sm.wmax[q]);
        const unsigned long long t0 = __builtin_amdgcn_s_memrealtime();
        auto give_up = [&]() __attribute__((always_inline)) -> bool {
            if (__builtin_amdgcn_s_memrealtime() - t0 < 100000ull) { __builtin_amdgcn_s_sleep(1); return false; }
            const unsigned tm = __builtin_amdgcn_readfirstlane(
                __builtin_amdgcn_raw_buffer_load_b32(rws, 0, OFF_TMO, AUX_AGT));
            if (tm == seq || __builtin_amdgcn_s_memrealtime() - t0 > DEADLINE_TICKS) {
                if (lane == 0) {
                    __builtin_amdgcn_raw_buffer_store_b32(seq, rws, 0, OFF_TMO, AUX_AGT);
                    __hip_atomic_store(&sm.fail, 1, __ATOMIC_RELAXED, __HIP_MEMORY_SCOPE_WORKGROUP);
                }
                return true;
            }
            return false;
        };
        if (jb != 0) {
            if (wave == 0 && lane == 0) {
                u32x2 gq; gq[0] = __builtin_bit_cast(unsigned, m); gq[1] = seq;
                __builtin_amdgcn_raw_buffer_store_b64(gq, rws, 0, OFF_AM + (unsigned)((b & 7) * 32 + jb) * 8u, AUX_AGT);
            }
        } else if (wave == 1) {
            const unsigned vg = (lane >= 1 && lane < 32) ? (unsigned)lane * 8u : OOB;
            float sm_ = m;
            for (;;) {
                const u32x2 gq = __builtin_amdgcn_raw_buffer_load_b64(rws, vg, OFF_AM + (unsigned)(b & 7) * 256u, AUX_AGT);
                sm_ = fmaxf(m, __builtin_bit_cast(float, gq[0]));
                if (__all((gq[1] == seq) | (vg == OOB))) break;
                if (give_up()) break;
            }
            sm_ = wave_max(sm_);
            u32x2 gq; gq[0] = __builtin_bit_cast(unsigned, sm_); gq[1] = seq;
#pragma unroll
            for (int q = 0; q < NRK; ++q) {
                const rsrc_t rq = mkrsrc(P.ws[q], WS_USED);
                if (lane == 0)
                    __builtin_amdgcn_raw_buffer_store_b64(gq, rq, 0, OFF_AM + 4096u + (unsigned)(rank * 8 + (b & 7)) * 8u, AUX_SYS);
            }
        }
        if (wave == 0) {
            float gm = 0.f;
            for (;;) {
                const u32x2 gq = __builtin_amdgcn_raw_buffer_load_b64(rws, (unsigned)lane * 8u, OFF_AM + 4096u, AUX_SYS);
                gm = __builtin_bit_cast(float, gq[0]);
                if (__all(gq[1] == seq)) break;
                if (give_up()) break;
            }
            gm = wave_max(gm);
            if (lane == 0) sm.gmax[0] = gm;
        }
    }
    __syncthreads();
    if (wave == 0) STAMP(8);

    const float amax = sm.gmax[0];
    const unsigned tmw = __builtin_amdgcn_readfirstlane(__builtin_amdgcn_raw_buffer_load_b32(rws, 0, OFF_TMO, AUX_AGT));
    const bool failed = (sm.fail != 0) | (tmw == seq);
    const float inv = amax > 0.f ? 448.0f / amax : 0.f;
    const rsrc_t ro = mkrsrc(P.out, 512u * 4096u);
#pragma unroll
    for (int r = 0; r < 4; ++r) {
        const float q0 = fminf(v[0][r] * inv, 448.f), q1 = fminf(v[1][r] * inv, 448.f);
        const float q2 = fminf(v[2][r] * inv, 448.f), q3 = fminf(v[3][r] * inv, 448.f);
        int pk = __builtin_amdgcn_cvt_pk_fp8_f32(q0, q1, 0, false);
        pk = __builtin_amdgcn_cvt_pk_fp8_f32(q2, q3, pk, true);
        if (failed) pk = 0x7f7f7f7f;
        const unsigned row = (unsigned)(mh * 256 + g * 64 + (wave >> 1) * 16 + (lane >> 4) * 4 + r);
        __builtin_amdgcn_raw_buffer_store_b32((unsigned)pk, ro, row * 4096u + (unsigned)(lane & 15) * 4u,
                                              (unsigned)(ntile * 128 + (wave & 1) * 64), 0);
    }
    if (tid == 0) __builtin_amdgcn_raw_buffer_store_b32(seq, rws, 0, OFF_CNT + b * 128u, AUX_AGT);
}

}

extern "C" void kernel_launch(int rank, int nranks, ncclComm_t comm,
                              hipStream_t stream,
                              void* const* d_in, const int* in_sizes, int n_in,
                              void* d_out, int out_size,
                              const int* in_dtypes, int out_dtype,
                              void* const* d_out_all,
                              void* const* d_ws_all, size_t ws_bytes) {
    if (nranks != NRK || (unsigned)rank >= (unsigned)NRK || ws_bytes < WS_USED) {
        fprintf(stderr, "unexpected configuration\n");
        return;
    }
    Params P;
    P.x = (const float*)d_in[0];
    P.w = (const float*)d_in[1];
    P.out = (unsigned char*)d_out;
    for (int q = 0; q < NRK; ++q) P.ws[q] = (char*)d_ws_all[q];
    P.myws = (char*)d_ws_all[rank];
    P.rank = rank;
    a2a_gemm_kernel<<<dim3(NWG), dim3(512), 0, stream>>>(P);
}
```

```cpp
#include <hip/hip_runtime.h>
#include <rccl/rccl.h>
#include <cstdio>
#include <cstdint>

typedef float    f32x4 __attribute__((ext_vector_type(4)));
typedef float    f32x2 __attribute__((ext_vector_type(2)));
typedef unsigned u32x4 __attribute__((ext_vector_type(4)));
typedef unsigned u32x2 __attribute__((ext_vector_type(2)));
typedef int      i32x4 __attribute__((ext_vector_type(4)));
typedef __bf16   bf16x8 __attribute__((ext_vector_type(8)));
typedef __bf16   bf16x2 __attribute__((ext_vector_type(2)));

#define UNTAG 1
#define LATEX 1

namespace {

constexpr int NRK   = 8;
constexpr int NWG   = 256;
constexpr int NUN   = 32;
constexpr int NU    = 12;

constexpr unsigned OFF_X   = 0;
constexpr unsigned OFF_XF  = 4u << 20;
constexpr unsigned OFF_AM  = OFF_XF + 16384u;
constexpr unsigned OFF_CNT = OFF_AM + 16384u;
constexpr unsigned OFF_RF  = OFF_CNT + 32768u;
constexpr unsigned OFF_TMO = OFF_RF + 65536u;
constexpr unsigned OFF_RED = 8u << 20;
constexpr unsigned WS_USED = OFF_RED + (24u << 20);

#define STAMP(i) do {} while (0)
#define VSTAMP(i, v) do {} while (0)
#define CLK() 0ull
constexpr int AUX_SYS = 17;
constexpr int AUX_AGT = 16;
constexpr unsigned long long DEADLINE_TICKS = 50000000ull;
constexpr unsigned OOB = 0x40000000u;

struct Params {
    const float* x;
    const float* w;
    unsigned char* out;
    char* myws;
    char* ws[NRK];
    int rank;
};

typedef __amdgpu_buffer_rsrc_t rsrc_t;

__device__ __forceinline__ rsrc_t mkrsrc(const void* p, unsigned bytes) {
    return __builtin_amdgcn_make_buffer_rsrc(const_cast<void*>(p), (short)0, (int)bytes, 0x00020000);
}
__device__ __forceinline__ unsigned pk2(float a, float b) {
    bf16x2 r = __builtin_convertvector((f32x2){a, b}, bf16x2);
    return __builtin_bit_cast(unsigned, r);
}
__device__ __forceinline__ float wave_max(float v) {
#pragma unroll
    for (int o = 32; o >= 1; o >>= 1) v = fmaxf(v, __shfl_xor(v, o, 64));
    return v;
}
template <int N> __device__ __forceinline__ void wait_vm() {
    asm volatile("s_waitcnt vmcnt(%0)" :: "i"(N) : "memory");
}
__device__ __forceinline__ void lds_set(int* p, int v) {
    __hip_atomic_store(p, v, __ATOMIC_RELAXED, __HIP_MEMORY_SCOPE_WORKGROUP);
}
__device__ __forceinline__ i32x4 lds_ld4(const int* p) {
    typedef const volatile __attribute__((address_space(3))) i32x4* lp_t;
    return *(lp_t)(p);
}
__device__ __forceinline__ int cons_done(const int* p) {
    const i32x4 v = lds_ld4(p);
    return __builtin_amdgcn_readfirstlane(min(min(v[0], v[1]), min(v[2], v[3])));
}

__device__ __forceinline__ int stage_of(int c, int r) {
    const int g = c & 3, j = c >> 2;
    if (r == 0) return j;
    const int t = (c - r) & 7;
    return 2 + 2 * t + j - (((g < t) || (g == t && j == 1)) ? 1 : 0) - ((g + 4 < t) ? 1 : 0);
}

__device__ __forceinline__ u32x4 cvt8(const f32x4& a, const f32x4& b) {
    u32x4 d;
    d[0] = pk2(a[0], a[1]); d[1] = pk2(a[2], a[3]); d[2] = pk2(b[0], b[1]); d[3] = pk2(b[2], b[3]);
    return d;
}
struct Smem {
    u32x4 ring[NU][512];
    f32x4 own[32 * 64];
    int prog[8];
    int tab[16];
    int fail;
    float wmax[8], gmax;
};

__global__ void __launch_bounds__(512, 2) a2a_gemm_kernel(Params P) {
    __shared__ __attribute__((aligned(16))) Smem sm;

    const int tid  = threadIdx.x;
    const int lane = tid & 63;
    const int wave = __builtin_amdgcn_readfirstlane(tid >> 6);
    const int b    = blockIdx.x;
    const int rank = P.rank;

    const rsrc_t rws = mkrsrc(P.myws, WS_USED);
    const int jb    = b >> 3;
    const int g     = (b & 7) >> 1;
    const int ntile = (b & 1) * 16 + (jb >> 1);
    const int mh    = jb & 1;
    const int tile  = ntile * 2 + mh;

    const int d   = (b + (b >> 3)) & 7;
    const int ph  = b >> 6;
    const int pj  = (b >> 3) & 7;
    const int pr_ = (rank - d) & 7;
    const unsigned long long tk0 = __builtin_amdgcn_s_memrealtime();
    f32x4 xv[32];
    auto load_x = [&]() __attribute__((always_inline)) {
        const rsrc_t rx = mkrsrc(P.x, 4096u * 512u * 4u);
        const unsigned vox = (unsigned)(512 * d + 64 * pj + (lane & 15)) * 2048u + (unsigned)(lane >> 4) * 32u;
#pragma unroll
        for (int q = 0; q < 4; ++q) {
            const unsigned xo = (unsigned)((2 * ph + (q >> 1) + pr_) & 7) * 256u + (unsigned)(q & 1) * 128u;
#pragma unroll
            for (int mf = 0; mf < 4; ++mf) {
                xv[8 * q + 2 * mf]     = __builtin_bit_cast(f32x4, __builtin_amdgcn_raw_buffer_load_b128(rx, vox, xo + mf * 32768u, 0));
                xv[8 * q + 2 * mf + 1] = __builtin_bit_cast(f32x4, __builtin_amdgcn_raw_buffer_load_b128(rx, vox, xo + mf * 32768u + 16u, 0));
            }
        }
    };
    const bool early = (pr_ == 0) | (ph == 0);
    if (wave == 6 && early) load_x();
    unsigned seq = __builtin_amdgcn_raw_buffer_load_b32(rws, 0, OFF_CNT + b * 128u, AUX_AGT);
    if (tid < 8) sm.prog[tid] = 0;
    if (tid == 32) sm.fail = 0;
    if (tid >= 64 && tid < 80) {
        const int i = tid - 64, t = i >> 1, j = i & 1;
        const int r = (g + 4 * j - t) & 7;
        sm.tab[stage_of(g + 4 * j, r)] = 512 * ((rank + r) & 7) + 64 * (g + 4 * j);
    }
    seq = __builtin_amdgcn_readfirstlane(seq) + 1u;
    __syncthreads();
    const unsigned tag = ((seq & 0x7fffu) | 0x8000u) << 16;

    if (wave < 4) {
        f32x4 acc[4][8];
#pragma unroll
        for (int f = 0; f < 4; ++f)
#pragma unroll
            for (int i = 0; i < 8; ++i) acc[f][i] = (f32x4){0.f, 0.f, 0.f, 0.f};
        const unsigned vox = (unsigned)lane * 16u;
        const unsigned xbase = OFF_X + (unsigned)((mh * 4 + g) * 16) * 32768u + (unsigned)wave * 4096u;
        u32x4 a0[4], a1[4], a2[4], a3[4];
        bf16x8 fb[8];
        unsigned long long cw = 0;
        int rdw = 0, rdx = 0;
        auto need = [&](int nw, int nx_) __attribute__((always_inline)) {
            if (rdw < nw || rdx < nx_) {
                unsigned long long t_ = CLK();
                for (;;) {
                    const i32x4 v = lds_ld4(&sm.prog[4]);
                    rdw = __builtin_amdgcn_readfirstlane(min(v[0], v[1]));
                    rdx = __builtin_amdgcn_readfirstlane(v[2]);
                    if (rdw >= nw && rdx >= nx_) break;
                    __builtin_amdgcn_s_sleep(1);
                }
                cw += CLK() - t_;
            }
            asm volatile("" ::: "memory");
        };
        auto issueA = [&](u32x4 (&av_)[4], int n) __attribute__((always_inline)) {
            const unsigned sx = xbase + (unsigned)n * 16384u;
#pragma unroll
            for (int f = 0; f < 4; ++f)
                av_[f] = __builtin_amdgcn_raw_buffer_load_b128(rws, vox + f * 1024u, sx, AUX_SYS);
        };
        auto block = [&](const u32x4 (&ca)[4], u32x4 (&na)[4], int u, int nslot) __attribute__((always_inline)) {
            need(u + 2 < NUN ? u + 2 : NUN, u + 4 < NUN ? u + 4 : NUN);
            __builtin_amdgcn_sched_barrier(0);
            const unsigned sx = u + 3 < NUN ? xbase + (unsigned)(u + 3) * 16384u : OOB;
            const u32x4* rb = &sm.ring[nslot][lane];
            i32x4 fl_v;
#pragma unroll
            for (int i = 0; i < 8; ++i) {
#pragma unroll
                for (int f = 0; f < 4; ++f)
                    asm volatile("v_mfma_f32_16x16x32_bf16 %0, %1, %2, %0" : "+a"(acc[f][i]) : "v"(ca[f]), "v"(fb[i]));
                __builtin_amdgcn_sched_barrier(0);
                fb[i] = __builtin_bit_cast(bf16x8, rb[i * 64]);
                if (i < 4) na[i] = __builtin_amdgcn_raw_buffer_load_b128(rws, vox + i * 1024u, sx, AUX_SYS);
                if (i == 5) fl_v = lds_ld4(&sm.prog[4]);
                __builtin_amdgcn_sched_barrier(0);
            }
            lds_set(&sm.prog[wave], u + 1);
            rdw = __builtin_amdgcn_readfirstlane(min(fl_v[0], fl_v[1]));
            rdx = __builtin_amdgcn_readfirstlane(fl_v[2]);
        };
        need(0, 1); issueA(a0, 0);
        need(0, 2); issueA(a1, 1);
        need(0, 3); issueA(a2, 2);
        need(1, 0);
        if (wave == 0) STAMP(2);
#pragma unroll
        for (int i = 0; i < 8; ++i) fb[i] = __builtin_bit_cast(bf16x8, sm.ring[0][i * 64 + lane]);
        unsigned long long c0 = CLK();
        int slot = 0;
        for (int u = 0; u < NUN; u += 4) {
            if (wave == 0) { if (u == 8) STAMP(11); if (u == 16) STAMP(12); if (u == 24) STAMP(14); }
            block(a0, a3, u, slot + 1);
            block(a1, a0, u + 1, slot + 2);
            block(a2, a1, u + 2, slot + 3);
            slot += 4; if (slot == NU) slot = 0;
            block(a3, a2, u + 3, slot);
        }
        asm volatile("s_nop 15\n\ts_nop 15"
                     : "+a"(acc[0][6]), "+a"(acc[1][6]), "+a"(acc[2][6]), "+a"(acc[3][6]),
                       "+a"(acc[0][7]), "+a"(acc[1][7]), "+a"(acc[2][7]), "+a"(acc[3][7]) :: "memory");
        if (wave == 0) { VSTAMP(0, CLK() - c0); VSTAMP(1, cw); STAMP(3); }
        if (wave == g) {
#pragma unroll
            for (int f = 0; f < 4; ++f)
#pragma unroll
                for (int i = 0; i < 8; ++i) sm.own[(f * 8 + i) * 64 + lane] = acc[f][i];
        } else {
            const unsigned so = OFF_RED + (unsigned)((tile * 4 + wave) * 4 + g) * 24576u;
#pragma unroll
            for (int f = 0; f < 4; ++f)
#pragma unroll
                for (int ip = 0; ip < 4; ++ip)
                    __builtin_amdgcn_raw_buffer_store_b128(cvt8(acc[f][2 * ip], acc[f][2 * ip + 1]), rws, vox + ip * 1024u, so + f * 4096u, AUX_AGT);
            wait_vm<0>();
            if (lane == 0)
                __builtin_amdgcn_raw_buffer_store_b32(seq, rws, 0, OFF_RF + (unsigned)((tile * 4 + wave) * 4 + g) * 64u, AUX_AGT);
        }
    } else if (wave < 6) {
        __builtin_amdgcn_s_setprio(2);
        const int nh = wave - 4;
        const rsrc_t rw = mkrsrc(P.w, 4096u * 4096u * 4u);
        const unsigned vow = (unsigned)(lane >> 4) * (8u * 16384u) + (unsigned)(lane & 15) * 16u;
        const unsigned wcol = (unsigned)(ntile * 128 + nh * 64) * 4u;
        unsigned long long lw = 0, l0 = CLK();
        f32x4 wb[3][8];
        const int tabv = sm.tab[lane & 15];
        auto issueW = [&](f32x4 (&wv)[8], int u) __attribute__((always_inline)) {
            unsigned sw = OOB;
            if (u < NUN) sw = (unsigned)(__builtin_amdgcn_readlane(tabv, u >> 1) + 32 * (u & 1)) * 16384u + wcol;
#pragma unroll
            for (int j = 0; j < 8; ++j)
                wv[j] = __builtin_bit_cast(f32x4, __builtin_amdgcn_raw_buffer_load_b128(rw, vow, sw + (unsigned)j * 16384u, 0));
        };
        int cdone = 0;
        auto fill = [&](const f32x4 (&wv)[8], int slot, int u) __attribute__((always_inline)) {
            if (cdone < u - (NU - 1)) {
                unsigned long long t_ = CLK();
                while ((cdone = cons_done(&sm.prog[0])) < u - (NU - 1)) __builtin_amdgcn_s_sleep(1);
                lw += CLK() - t_;
            }
            asm volatile("" ::: "memory");
#pragma unroll
            for (int i = 0; i < 4; ++i) {
                u32x4 t;
                t[0] = pk2(wv[0][i], wv[1][i]);
                t[1] = pk2(wv[2][i], wv[3][i]);
                t[2] = pk2(wv[4][i], wv[5][i]);
                t[3] = pk2(wv[6][i], wv[7][i]);
                sm.ring[slot][(nh * 4 + i) * 64 + lane] = t;
            }
            asm volatile("" ::: "memory");
            lds_set(&sm.prog[4 + nh], u + 1);
            if (nh == 0 && u == 7) STAMP(4);
            if (nh == 0 && u == 15) STAMP(5);
            if (nh == 0 && u == 23) STAMP(6);
        };
#pragma unroll
        for (int dd = 0; dd < 3; ++dd) issueW(wb[dd], dd);
        int fslot = 0;
        for (int u = 0; u < NUN; u += 3) {
#pragma unroll
            for (int dd = 0; dd < 3; ++dd) {
                if (u + dd < NUN) fill(wb[dd], fslot + dd, u + dd);
                issueW(wb[dd], u + dd + 3);
            }
            fslot += 3; if (fslot == NU) fslot = 0;
        }
        if (nh == 0) { VSTAMP(3, CLK() - l0); VSTAMP(4, lw); STAMP(10); }
    } else if (wave == 6) {
        __builtin_amdgcn_s_setprio(2);
        unsigned so[2], fo[2];
#pragma unroll
        for (int t = 0; t < 2; ++t) {
            const int c = (2 * ph + t + pr_) & 7;
            const unsigned blk = (unsigned)(((pj >> 2) * 4 + (c & 3)) * 16 + stage_of(c, pr_));
            so[t] = OFF_X + blk * 32768u + (unsigned)(pj & 3) * 4096u;
            fo[t] = OFF_XF + (blk * 32u + (unsigned)(pj & 3) * 4u) * 4u;
        }
        char* dst = P.myws;
#pragma unroll
        for (int q = 0; q < NRK; ++q) if (q == d) dst = P.ws[q];
        const rsrc_t rdst = mkrsrc(dst, WS_USED);
#define PT1 400
#define PT2 800
#define PT3 1200
        if (pr_ != 0) {
            const unsigned long long tgo = ph == 0 ? 0 : ph == 1 ? PT1 : ph == 2 ? PT2 : PT3;
            if (!early) {
                while (__builtin_amdgcn_s_memrealtime() - tk0 + 150 < tgo) __builtin_amdgcn_s_sleep(2);
                load_x();
            }
            while (__builtin_amdgcn_s_memrealtime() - tk0 < tgo) __builtin_amdgcn_s_sleep(2);
        }
#pragma unroll
        for (int J = 0; J < 16; ++J)
            __builtin_amdgcn_raw_buffer_store_b128(cvt8(xv[2 * J], xv[2 * J + 1]), rdst, (unsigned)lane * 16u,
                                                   so[J >> 3] + ((J >> 2) & 1) * 16384u + (J & 3) * 1024u, AUX_SYS);
#pragma unroll
        for (int q = 0; q < 4; ++q) {
            if (q == 0) wait_vm<12>(); else if (q == 1) wait_vm<9>(); else if (q == 2) wait_vm<6>(); else wait_vm<3>();
            if (lane == 0) {
                u32x4 fl = {seq, seq, seq, seq};
                __builtin_amdgcn_raw_buffer_store_b128(fl, rdst, 0, fo[q >> 1] + (q & 1) * 64u, AUX_SYS);
            }
        }
        STAMP(1);
    } else {
        __builtin_amdgcn_s_setprio(2);
        const unsigned long long t0 = __builtin_amdgcn_s_memrealtime();
        const unsigned fo = OFF_XF + (unsigned)(mh * 4 + g) * 2048u;
        const unsigned xtouch = OFF_X + (unsigned)((mh * 4 + g) * 16) * 32768u + (unsigned)(ntile & 15) * 1024u;
        unsigned sink = 0;
        int av = 0;
        while (av < NUN) {
            const unsigned vf = (lane < 16 && av + (lane >> 1) < NUN) ? (unsigned)lane * 32u : OOB;
            const unsigned sf = fo + (unsigned)av * 64u;
            u32x4 f0 = __builtin_amdgcn_raw_buffer_load_b128(rws, vf, sf, AUX_SYS);
            u32x4 f1 = __builtin_amdgcn_raw_buffer_load_b128(rws, vf + 16u, sf, AUX_SYS);
            const bool ok = (f0[0] == seq) & (f0[1] == seq) & (f0[2] == seq) & (f0[3] == seq) &
                            (f1[0] == seq) & (f1[1] == seq) & (f1[2] == seq) & (f1[3] == seq);
            const unsigned m = (unsigned)__ballot(ok) & 0xffffu;
            const unsigned m2 = m & (m >> 1);
            const unsigned xm = (~m2 & 0x5555u) | 0x10000u;
            const int na = __builtin_amdgcn_readfirstlane(av + (__builtin_ctz(xm) >> 1));
            if (na > av) {
                if (av < 4 && na >= 4) STAMP(13);
                if (na >= NUN) STAMP(15);
                {
                    const unsigned vt = (av + (lane >> 3) < na) ? (unsigned)(lane >> 3) * 16384u + (unsigned)(lane & 7) * 128u : OOB;
                    sink ^= __builtin_amdgcn_raw_buffer_load_b32(rws, vt, xtouch + (unsigned)av * 16384u, 0);
                }
                av = na;
                lds_set(&sm.prog[6], av);
            } else if (__builtin_amdgcn_s_memrealtime() - t0 <= 100000ull) {
                __builtin_amdgcn_s_sleep(2);
            } else {
                const unsigned tm = __builtin_amdgcn_readfirstlane(
                    __builtin_amdgcn_raw_buffer_load_b32(rws, 0, OFF_TMO, AUX_AGT));
                if (tm == seq || __builtin_amdgcn_s_memrealtime() - t0 > DEADLINE_TICKS) {
                    if (lane == 0) {
                        __builtin_amdgcn_raw_buffer_store_b32(seq, rws, 0, OFF_TMO, AUX_AGT);
                        __hip_atomic_store(&sm.fail, 1, __ATOMIC_RELAXED, __HIP_MEMORY_SCOPE_WORKGROUP);
                    }
                    av = NUN;
                    lds_set(&sm.prog[6], av);
                }
            }
            av = __builtin_amdgcn_readfirstlane(av);
        }
        if (sink == 0x9e3779b9u && seq == 0u) sm.fail = 1;
    }
    __builtin_amdgcn_s_setprio(0);
    __syncthreads();

    const unsigned long long tw0 = __builtin_amdgcn_s_memrealtime();
    auto give_up = [&]() __attribute__((always_inline)) -> bool {
        if (__builtin_amdgcn_s_memrealtime() - tw0 < 100000ull) { __builtin_amdgcn_s_sleep(1); return false; }
        const unsigned tm = __builtin_amdgcn_readfirstlane(
            __builtin_amdgcn_raw_buffer_load_b32(rws, 0, OFF_TMO, AUX_AGT));
        if (tm == seq || __builtin_amdgcn_s_memrealtime() - tw0 > DEADLINE_TICKS) {
            if (lane == 0) {
                __builtin_amdgcn_raw_buffer_store_b32(seq, rws, 0, OFF_TMO, AUX_AGT);
                __hip_atomic_store(&sm.fail, 1, __ATOMIC_RELAXED, __HIP_MEMORY_SCOPE_WORKGROUP);
            }
            return true;
        }
        return false;
    };

    const int fbase = (wave >> 1) * 8 + (wave & 1) * 4;
    f32x4 v[4];
    {
        for (;;) {
            const unsigned f = __builtin_amdgcn_raw_buffer_load_b32(rws, (unsigned)(lane & 3) * 64u,
                                                                    OFF_RF + (unsigned)(tile * 4 + g) * 256u, AUX_AGT);
            if (__all((f == seq) | ((lane & 3) == g))) break;
            if (give_up()) break;
        }
        asm volatile("" ::: "memory");
        u32x4 pr[3][2];
#pragma unroll
        for (int s = 0; s < 3; ++s) {
            const int src = s + (s >= g ? 1 : 0);
            const unsigned so = OFF_RED + (unsigned)((tile * 4 + g) * 4 + src) * 24576u + (unsigned)wave * 2048u;
#pragma unroll
            for (int i = 0; i < 2; ++i)
                pr[s][i] = __builtin_amdgcn_raw_buffer_load_b128(rws, (unsigned)lane * 16u + i * 1024u, so, AUX_AGT);
        }
#pragma unroll
        for (int i = 0; i < 4; ++i) v[i] = sm.own[(fbase + i) * 64 + lane];
#pragma unroll
        for (int s = 0; s < 3; ++s)
#pragma unroll
            for (int i = 0; i < 4; ++i) {
                const unsigned lo = pr[s][i >> 1][(i & 1) * 2], hi = pr[s][i >> 1][(i & 1) * 2 + 1];
                v[i][0] += __builtin_bit_cast(float, lo << 16);
                v[i][1] += __builtin_bit_cast(float, lo & 0xffff0000u);
                v[i][2] += __builtin_bit_cast(float, hi << 16);
                v[i][3] += __builtin_bit_cast(float, hi & 0xffff0000u);
            }
    }
    float vmax = 0.f;
#pragma unroll
    for (int i = 0; i < 4; ++i)
#pragma unroll
        for (int r = 0; r < 4; ++r) { v[i][r] = fmaxf(v[i][r], 0.f); vmax = fmaxf(vmax, v[i][r]); }
    vmax = wave_max(vmax);
    if (lane == 0) sm.wmax[wave] = vmax;
    __syncthreads();
    if (wave == 0) STAMP(7);

    {
        float m = sm.wmax[0];
#pragma unroll
        for (int q = 1; q < 8; ++q) m = fmaxf(m, sm.wmax[q]);
        if (jb != 0) {
            if (wave == 1 && lane == 0) {
                u32x2 gq; gq[0] = __builtin_bit_cast(unsigned, m); gq[1] = seq;
                __builtin_amdgcn_raw_buffer_store_b64(gq, rws, 0, OFF_AM + (unsigned)((b & 7) * 32 + jb) * 8u, AUX_AGT);
            }
        } else if (wave == 1) {
            const unsigned vg = (lane >= 1 && lane < 32) ? (unsigned)lane * 8u : OOB;
            float sm_ = m;
            for (;;) {
                const u32x2 gq = __builtin_amdgcn_raw_buffer_load_b64(rws, vg, OFF_AM + (unsigned)(b & 7) * 256u, AUX_AGT);
                sm_ = fmaxf(m, __builtin_bit_cast(float, gq[0]));
                if (__all((gq[1] == seq) | (vg == OOB))) break;
                if (give_up()) break;
            }
            sm_ = wave_max(sm_);
            u32x2 gq; gq[0] = __builtin_bit_cast(unsigned, sm_); gq[1] = seq;
#pragma unroll
            for (int q = 0; q < NRK; ++q) {
                const rsrc_t rq = mkrsrc(P.ws[q], WS_USED);
                if (lane == 0)
                    __builtin_amdgcn_raw_buffer_store_b64(gq, rq, 0, OFF_AM + 4096u + (unsigned)(rank * 8 + (b & 7)) * 8u, AUX_SYS);
            }
        }
        if (wave == 0) {
            float gm = 0.f;
            for (;;) {
                const u32x2 gq = __builtin_amdgcn_raw_buffer_load_b64(rws, (unsigned)lane * 8u, OFF_AM + 4096u, AUX_SYS);
                gm = __builtin_bit_cast(float, gq[0]);
                if (__all(gq[1] == seq)) break;
                if (give_up()) break;
            }
            gm = wave_max(gm);
            if (lane == 0) sm.gmax = gm;
        }
    }
    __syncthreads();
    if (wave == 0) STAMP(8);

    const float amax = sm.gmax;
    const unsigned tmw = __builtin_amdgcn_readfirstlane(__builtin_amdgcn_raw_buffer_load_b32(rws, 0, OFF_TMO, AUX_AGT));
    const bool failed = (sm.fail != 0) | (tmw == seq);
    const float inv = amax > 0.f ? 448.0f / amax : 0.f;
    const rsrc_t ro = mkrsrc(P.out, 512u * 4096u);
#pragma unroll
    for (int r = 0; r < 4; ++r) {
        const float q0 = fminf(v[0][r] * inv, 448.f), q1 = fminf(v[1][r] * inv, 448.f);
        const float q2 = fminf(v[2][r] * inv, 448.f), q3 = fminf(v[3][r] * inv, 448.f);
        int pk = __builtin_amdgcn_cvt_pk_fp8_f32(q0, q1, 0, false);
        pk = __builtin_amdgcn_cvt_pk_fp8_f32(q2, q3, pk, true);
        if (failed) pk = 0x7f7f7f7f;
        const unsigned row = (unsigned)(mh * 256 + g * 64 + (wave >> 1) * 16 + (lane >> 4) * 4 + r);
        __builtin_amdgcn_raw_buffer_store_b32((unsigned)pk, ro, row * 4096u + (unsigned)(lane & 15) * 4u,
                                              (unsigned)(ntile * 128 + (wave & 1) * 64), 0);
    }
    if (tid == 0) __builtin_amdgcn_raw_buffer_store_b32(seq, rws, 0, OFF_CNT + b * 128u, AUX_AGT);
}

}

extern "C" void kernel_launch(int rank, int nranks, ncclComm_t comm,
                              hipStream_t stream,
                              void* const* d_in, const int* in_sizes, int n_in,
                              void* d_out, int out_size,
                              const int* in_dtypes, int out_dtype,
                              void* const* d_out_all,
                              void* const* d_ws_all, size_t ws_bytes) {
    if (nranks != NRK || (unsigned)rank >= (unsigned)NRK || ws_bytes < WS_USED) {
        fprintf(stderr, "unexpected configuration\n");
        return;
    }
    Params P;
    P.x = (const float*)d_in[0];
    P.w = (const float*)d_in[1];
    P.out = (unsigned char*)d_out;
    for (int q = 0; q < NRK; ++q) P.ws[q] = (char*)d_ws_all[q];
    P.myws = (char*)d_ws_all[rank];
    P.rank = rank;
    a2a_gemm_kernel<<<dim3(NWG), dim3(512), 0, stream>>>(P);
}
```

```cpp
#include <hip/hip_runtime.h>
#include <rccl/rccl.h>
#include <cstdio>
#include <cstdint>

typedef float    f32x4 __attribute__((ext_vector_type(4)));
typedef float    f32x2 __attribute__((ext_vector_type(2)));
typedef unsigned u32x4 __attribute__((ext_vector_type(4)));
typedef unsigned u32x2 __attribute__((ext_vector_type(2)));
typedef int      i32x4 __attribute__((ext_vector_type(4)));
typedef __bf16   bf16x8 __attribute__((ext_vector_type(8)));
typedef __bf16   bf16x2 __attribute__((ext_vector_type(2)));

#define UNTAG 1
#define LATEX 1

namespace {

constexpr int NRK   = 8;
constexpr int NWG   = 256;
constexpr int NUN   = 32;
constexpr int NU    = 12;

constexpr unsigned OFF_X   = 0;
constexpr unsigned OFF_XF  = 4u << 20;
constexpr unsigned OFF_AM  = OFF_XF + 16384u;
constexpr unsigned OFF_CNT = OFF_AM + 16384u;
constexpr unsigned OFF_RF  = OFF_CNT + 32768u;
constexpr unsigned OFF_TMO = OFF_RF + 65536u;
constexpr unsigned OFF_RED = 8u << 20;
constexpr unsigned WS_USED = OFF_RED + (24u << 20);

#define STAMP(i) do {} while (0)
#define VSTAMP(i, v) do {} while (0)
#define CLK() 0ull
constexpr int AUX_SYS = 17;
constexpr int AUX_AGT = 16;
constexpr unsigned long long DEADLINE_TICKS = 50000000ull;
constexpr unsigned OOB = 0x40000000u;

struct Params {
    const float* x;
    const float* w;
    unsigned char* out;
    char* myws;
    char* ws[NRK];
    int rank;
};

typedef __amdgpu_buffer_rsrc_t rsrc_t;

__device__ __forceinline__ rsrc_t mkrsrc(const void* p, unsigned bytes) {
    return __builtin_amdgcn_make_buffer_rsrc(const_cast<void*>(p), (short)0, (int)bytes, 0x00020000);
}
__device__ __forceinline__ unsigned pk2(float a, float b) {
    bf16x2 r = __builtin_convertvector((f32x2){a, b}, bf16x2);
    return __builtin_bit_cast(unsigned, r);
}
__device__ __forceinline__ float wave_max(float v) {
#pragma unroll
    for (int o = 32; o >= 1; o >>= 1) v = fmaxf(v, __shfl_xor(v, o, 64));
    return v;
}
template <int N> __device__ __forceinline__ void wait_vm() {
    asm volatile("s_waitcnt vmcnt(%0)" :: "i"(N) : "memory");
}
__device__ __forceinline__ void lds_set(int* p, int v) {
    __hip_atomic_store(p, v, __ATOMIC_RELAXED, __HIP_MEMORY_SCOPE_WORKGROUP);
}
__device__ __forceinline__ i32x4 lds_ld4(const int* p) {
    typedef const volatile __attribute__((address_space(3))) i32x4* lp_t;
    return *(lp_t)(p);
}
__device__ __forceinline__ int lds_ld1(const int* p) {
    typedef const volatile __attribute__((address_space(3))) int* lp_t;
    return *(lp_t)(p);
}
__device__ __forceinline__ int cons_done(const int* p) {
    const i32x4 v = lds_ld4(p);
    return __builtin_amdgcn_readfirstlane(min(min(v[0], v[1]), min(v[2], v[3])));
}

__device__ __forceinline__ int stage_of(int c, int r) {
    const int g = c & 3, j = c >> 2;
    if (r == 0) return j;
    const int t = (c - r) & 7;
    return 2 + 2 * t + j - (((g < t) || (g == t && j == 1)) ? 1 : 0) - ((g + 4 < t) ? 1 : 0);
}

__device__ __forceinline__ u32x4 cvt8(const f32x4& a, const f32x4& b) {
    u32x4 d;
    d[0] = pk2(a[0], a[1]); d[1] = pk2(a[2], a[3]); d[2] = pk2(b[0], b[1]); d[3] = pk2(b[2], b[3]);
    return d;
}
struct Smem {
    u32x4 ring[NU][512];
    f32x4 own[32 * 64];
    int prog[8];
    int tab[16];
    int plan[16];
    int fail;
    float wmax[8], gmax;
};

__global__ void __launch_bounds__(512, 2) a2a_gemm_kernel(Params P) {
    __shared__ __attribute__((aligned(16))) Smem sm;

    const int tid  = threadIdx.x;
    const int lane = tid & 63;
    const int wave = __builtin_amdgcn_readfirstlane(tid >> 6);
    const int b    = blockIdx.x;
    const int rank = P.rank;

    const rsrc_t rws = mkrsrc(P.myws, WS_USED);
    const int jb    = b >> 3;
    const int g     = (b & 7) >> 1;
    const int ntile = (b & 1) * 16 + (jb >> 1);
    const int mh    = jb & 1;
    const int tile  = ntile * 2 + mh;

    const int d   = (b + (b >> 3)) & 7;
    const int ph  = b >> 6;
    const int pj  = (b >> 3) & 7;
    const int pr_ = (rank - d) & 7;
    const unsigned long long tk0 = __builtin_amdgcn_s_memrealtime();
    f32x4 xv[32];
    auto load_x = [&]() __attribute__((always_inline)) {
        const rsrc_t rx = mkrsrc(P.x, 4096u * 512u * 4u);
        const unsigned vox = (unsigned)(512 * d + 64 * pj + (lane & 15)) * 2048u + (unsigned)(lane >> 4) * 32u;
#pragma unroll
        for (int q = 0; q < 4; ++q) {
            const unsigned xo = (unsigned)((2 * ph + (q >> 1) + pr_) & 7) * 256u + (unsigned)(q & 1) * 128u;
#pragma unroll
            for (int mf = 0; mf < 4; ++mf) {
                xv[8 * q + 2 * mf]     = __builtin_bit_cast(f32x4, __builtin_amdgcn_raw_buffer_load_b128(rx, vox, xo + mf * 32768u, 0));
                xv[8 * q + 2 * mf + 1] = __builtin_bit_cast(f32x4, __builtin_amdgcn_raw_buffer_load_b128(rx, vox, xo + mf * 32768u + 16u, 0));
            }
        }
    };
    const bool early = (pr_ == 0) | (ph == 0);
    if (wave == 6 && early) load_x();
    unsigned seq = __builtin_amdgcn_raw_buffer_load_b32(rws, 0, OFF_CNT + b * 128u, AUX_AGT);
    if (tid < 8) sm.prog[tid] = 0;
    if (tid == 32) sm.fail = 0;
    if (tid >= 64 && tid < 80) {
        const int i = tid - 64, t = i >> 1, j = i & 1;
        const int r = (g + 4 * j - t) & 7;
        sm.tab[stage_of(g + 4 * j, r)] = 512 * ((rank + r) & 7) + 64 * (g + 4 * j);
    }
    seq = __builtin_amdgcn_readfirstlane(seq) + 1u;
    __syncthreads();
    const unsigned tag = ((seq & 0x7fffu) | 0x8000u) << 16;

    if (wave < 4) {
        f32x4 acc[4][8];
#pragma unroll
        for (int f = 0; f < 4; ++f)
#pragma unroll
            for (int i = 0; i < 8; ++i) acc[f][i] = (f32x4){0.f, 0.f, 0.f, 0.f};
        const unsigned vox = (unsigned)lane * 16u;
        const unsigned xbase = OFF_X + (unsigned)((mh * 4 + g) * 16) * 32768u + (unsigned)wave * 4096u;
        u32x4 a0[4], a1[4], a2[4], a3[4];
        bf16x8 fb[8];
        unsigned long long cw = 0;
        int rdw = 0, rdx = 0;
        int plv = 0;
        int csn = 0;
        auto need = [&](int nw, int nx_) __attribute__((always_inline)) -> bool {
            bool slow = false;
            if (rdw < nw || rdx < nx_) {
                slow = true;
                unsigned long long t_ = CLK();
                for (;;) {
                    const i32x4 v = lds_ld4(&sm.prog[4]);
                    plv = lds_ld1(&sm.plan[lane & 15]);
                    rdw = __builtin_amdgcn_readfirstlane(min(v[0], v[1]));
                    rdx = __builtin_amdgcn_readfirstlane(v[2]);
                    if (rdw >= nw && rdx >= nx_) break;
                    __builtin_amdgcn_s_sleep(1);
                }
                cw += CLK() - t_;
            }
            asm volatile("" ::: "memory");
            return slow;
        };
        auto issueA = [&](u32x4 (&av_)[4], int n) __attribute__((always_inline)) {
            const unsigned sx = xbase + (unsigned)(__builtin_amdgcn_readlane(plv, n >> 1) * 2 + (n & 1)) * 16384u;
#pragma unroll
            for (int f = 0; f < 4; ++f)
                av_[f] = __builtin_amdgcn_raw_buffer_load_b128(rws, vox + f * 1024u, sx, AUX_SYS);
        };
        auto block = [&](const u32x4 (&ca)[4], u32x4 (&na)[4], int u, int nslot) __attribute__((always_inline)) {
            if (need(u + 2 < NUN ? u + 2 : NUN, u + 4 < NUN ? u + 4 : NUN))
                csn = __builtin_amdgcn_readlane(plv, ((u + 3) >> 1) & 15);
            __builtin_amdgcn_sched_barrier(0);
            const unsigned sx = u + 3 < NUN ? xbase + (unsigned)(csn * 2 + ((u + 3) & 1)) * 16384u : OOB;
            const u32x4* rb = &sm.ring[nslot][lane];
            i32x4 fl_v;
#pragma unroll
            for (int i = 0; i < 8; ++i) {
#pragma unroll
                for (int f = 0; f < 4; ++f)
                    asm volatile("v_mfma_f32_16x16x32_bf16 %0, %1, %2, %0" : "+a"(acc[f][i]) : "v"(ca[f]), "v"(fb[i]));
                __builtin_amdgcn_sched_barrier(0);
                fb[i] = __builtin_bit_cast(bf16x8, rb[i * 64]);
                if (i < 4) na[i] = __builtin_amdgcn_raw_buffer_load_b128(rws, vox + i * 1024u, sx, AUX_SYS);
                if (i == 5) { fl_v = lds_ld4(&sm.prog[4]); plv = lds_ld1(&sm.plan[lane & 15]); }
                __builtin_amdgcn_sched_barrier(0);
            }
            lds_set(&sm.prog[wave], u + 1);
            rdw = __builtin_amdgcn_readfirstlane(min(fl_v[0], fl_v[1]));
            rdx = __builtin_amdgcn_readfirstlane(fl_v[2]);
            csn = __builtin_amdgcn_readlane(plv, ((u + 4) >> 1) & 15);
        };
        need(0, 1); issueA(a0, 0);
        need(0, 2); issueA(a1, 1);
        need(0, 3); issueA(a2, 2);
        need(1, 0);
        csn = __builtin_amdgcn_readlane(plv, 1);
        if (wave == 0) STAMP(2);
#pragma unroll
        for (int i = 0; i < 8; ++i) fb[i] = __builtin_bit_cast(bf16x8, sm.ring[0][i * 64 + lane]);
        unsigned long long c0 = CLK();
        int slot = 0;
        for (int u = 0; u < NUN; u += 4) {
            if (wave == 0) { if (u == 8) STAMP(11); if (u == 16) STAMP(12); if (u == 24) STAMP(14); }
            block(a0, a3, u, slot + 1);
            block(a1, a0, u + 1, slot + 2);
            block(a2, a1, u + 2, slot + 3);
            slot += 4; if (slot == NU) slot = 0;
            block(a3, a2, u + 3, slot);
        }
        asm volatile("s_nop 15\n\ts_nop 15"
                     : "+a"(acc[0][6]), "+a"(acc[1][6]), "+a"(acc[2][6]), "+a"(acc[3][6]),
                       "+a"(acc[0][7]), "+a"(acc[1][7]), "+a"(acc[2][7]), "+a"(acc[3][7]) :: "memory");
        if (wave == 0) { VSTAMP(0, CLK() - c0); VSTAMP(1, cw); STAMP(3); }
        if (wave == g) {
#pragma unroll
            for (int f = 0; f < 4; ++f)
#pragma unroll
                for (int i = 0; i < 8; ++i) sm.own[(f * 8 + i) * 64 + lane] = acc[f][i];
        } else {
            const unsigned so = OFF_RED + (unsigned)((tile * 4 + wave) * 4 + g) * 24576u;
#pragma unroll
            for (int f = 0; f < 4; ++f)
#pragma unroll
                for (int ip = 0; ip < 4; ++ip)
                    __builtin_amdgcn_raw_buffer_store_b128(cvt8(acc[f][2 * ip], acc[f][2 * ip + 1]), rws, vox + ip * 1024u, so + f * 4096u, AUX_AGT);
            wait_vm<0>();
            if (lane == 0)
                __builtin_amdgcn_raw_buffer_store_b32(seq, rws, 0, OFF_RF + (unsigned)((tile * 4 + wave) * 4 + g) * 64u, AUX_AGT);
        }
    } else if (wave < 6) {
        __builtin_amdgcn_s_setprio(2);
        const int nh = wave - 4;
        const rsrc_t rw = mkrsrc(P.w, 4096u * 4096u * 4u);
        const unsigned vow = (unsigned)(lane >> 4) * (8u * 16384u) + (unsigned)(lane & 15) * 16u;
        const unsigned wcol = (unsigned)(ntile * 128 + nh * 64) * 4u;
        unsigned long long lw = 0, l0 = CLK();
        f32x4 wb[3][8];
        const int tabv = sm.tab[lane & 15];
        int planned = 0, plv = 0;
        auto issueW = [&](f32x4 (&wv)[8], int u) __attribute__((always_inline)) {
            unsigned sw = OOB;
            if (u < NUN) {
                if (planned <= (u >> 1)) {
                    while ((planned = __builtin_amdgcn_readfirstlane(lds_ld1(&sm.prog[7]))) <= (u >> 1)) __builtin_amdgcn_s_sleep(1);
                    plv = lds_ld1(&sm.plan[lane & 15]);
                }
                const int cs = __builtin_amdgcn_readlane(plv, u >> 1);
                sw = (unsigned)(__builtin_amdgcn_readlane(tabv, cs) + 32 * (u & 1)) * 16384u + wcol;
            }
#pragma unroll
            for (int j = 0; j < 8; ++j)
                wv[j] = __builtin_bit_cast(f32x4, __builtin_amdgcn_raw_buffer_load_b128(rw, vow, sw + (unsigned)j * 16384u, 0));
        };
        int cdone = 0;
        auto fill = [&](const f32x4 (&wv)[8], int slot, int u) __attribute__((always_inline)) {
            if (cdone < u - (NU - 1)) {
                unsigned long long t_ = CLK();
                while ((cdone = cons_done(&sm.prog[0])) < u - (NU - 1)) __builtin_amdgcn_s_sleep(1);
                lw += CLK() - t_;
            }
            asm volatile("" ::: "memory");
#pragma unroll
            for (int i = 0; i < 4; ++i) {
                u32x4 t;
                t[0] = pk2(wv[0][i], wv[1][i]);
                t[1] = pk2(wv[2][i], wv[3][i]);
                t[2] = pk2(wv[4][i], wv[5][i]);
                t[3] = pk2(wv[6][i], wv[7][i]);
                sm.ring[slot][(nh * 4 + i) * 64 + lane] = t;
            }
            asm volatile("" ::: "memory");
            lds_set(&sm.prog[4 + nh], u + 1);
            if (nh == 0 && u == 7) STAMP(4);
            if (nh == 0 && u == 15) STAMP(5);
            if (nh == 0 && u == 23) STAMP(6);
        };
#pragma unroll
        for (int dd = 0; dd < 3; ++dd) issueW(wb[dd], dd);
        int fslot = 0;
        for (int u = 0; u < NUN; u += 3) {
#pragma unroll
            for (int dd = 0; dd < 3; ++dd) {
                if (u + dd < NUN) fill(wb[dd], fslot + dd, u + dd);
                issueW(wb[dd], u + dd + 3);
            }
            fslot += 3; if (fslot == NU) fslot = 0;
        }
        if (nh == 0) { VSTAMP(3, CLK() - l0); VSTAMP(4, lw); STAMP(10); }
    } else if (wave == 6) {
        __builtin_amdgcn_s_setprio(2);
        unsigned so[2], fo[2];
#pragma unroll
        for (int t = 0; t < 2; ++t) {
            const int c = (2 * ph + t + pr_) & 7;
            const unsigned blk = (unsigned)(((pj >> 2) * 4 + (c & 3)) * 16 + stage_of(c, pr_));
            so[t] = OFF_X + blk * 32768u + (unsigned)(pj & 3) * 4096u;
            fo[t] = OFF_XF + (blk * 32u + (unsigned)(pj & 3) * 4u) * 4u;
        }
        char* dst = P.myws;
#pragma unroll
        for (int q = 0; q < NRK; ++q) if (q == d) dst = P.ws[q];
        const rsrc_t rdst = mkrsrc(dst, WS_USED);
#define PT1 400
#define PT2 800
#define PT3 1200
        if (pr_ != 0) {
            const unsigned long long tgo = ph == 0 ? 0 : ph == 1 ? PT1 : ph == 2 ? PT2 : PT3;
            if (!early) {
                while (__builtin_amdgcn_s_memrealtime() - tk0 + 150 < tgo) __builtin_amdgcn_s_sleep(2);
                load_x();
            }
            while (__builtin_amdgcn_s_memrealtime() - tk0 < tgo) __builtin_amdgcn_s_sleep(2);
        }
#pragma unroll
        for (int J = 0; J < 16; ++J)
            __builtin_amdgcn_raw_buffer_store_b128(cvt8(xv[2 * J], xv[2 * J + 1]), rdst, (unsigned)lane * 16u,
                                                   so[J >> 3] + ((J >> 2) & 1) * 16384u + (J & 3) * 1024u, AUX_SYS);
#pragma unroll
        for (int q = 0; q < 4; ++q) {
            if (q == 0) wait_vm<12>(); else if (q == 1) wait_vm<9>(); else if (q == 2) wait_vm<6>(); else wait_vm<3>();
            if (lane == 0) {
                u32x4 fl = {seq, seq, seq, seq};
                __builtin_amdgcn_raw_buffer_store_b128(fl, rdst, 0, fo[q >> 1] + (q & 1) * 64u, AUX_SYS);
            }
        }
        STAMP(1);
    } else {
        __builtin_amdgcn_s_setprio(2);
        const unsigned long long t0 = __builtin_amdgcn_s_memrealtime();
        const unsigned fo = OFF_XF + (unsigned)(mh * 4 + g) * 2048u;
        const unsigned xtouch = OFF_X + (unsigned)((mh * 4 + g) * 16) * 32768u + (unsigned)(ntile & 15) * 1024u;
        unsigned sink = 0, tch = 0;
#define MINAHEAD 4
        unsigned have = 0, sched = 0;
        int np = 0, na_ = 0;
        unsigned long long planpk = 0;
        while (na_ < 16) {
            bool prog_ = false;
            unsigned newst = 0;
            if (have != 0xffffu) {
                u32x4 f0 = __builtin_amdgcn_raw_buffer_load_b128(rws, (unsigned)lane * 32u, fo, AUX_SYS);
                u32x4 f1 = __builtin_amdgcn_raw_buffer_load_b128(rws, (unsigned)lane * 32u + 16u, fo, AUX_SYS);
                const bool ok = (f0[0] == seq) & (f0[1] == seq) & (f0[2] == seq) & (f0[3] == seq) &
                                (f1[0] == seq) & (f1[1] == seq) & (f1[2] == seq) & (f1[3] == seq);
                const unsigned long long m = __ballot(ok);
                const unsigned long long m4 = m & (m >> 1) & (m >> 2) & (m >> 3);
                const unsigned nh_ = (unsigned)__ballot(lane < 16 && ((m4 >> (4 * (lane & 15))) & 1ull));
                const unsigned fresh = nh_ & ~have;
                if (fresh) {
                    newst = fresh;
                    have |= fresh; prog_ = true;
                }
            }
            const int cst = cons_done(&sm.prog[0]) >> 1;
            const int np0 = np;
            while (np < 16) {
                unsigned cand = have & ~sched;
                if (!cand) { if (np - cst >= MINAHEAD) break; cand = ~sched & 0xffffu; }
                const int sn = __builtin_ctz(cand);
                sched |= 1u << sn;
                planpk |= (unsigned long long)sn << (4 * np);
                lds_set(&sm.plan[np], sn);
                ++np;
            }
            if (np != np0) { asm volatile("" ::: "memory"); lds_set(&sm.prog[7], np); prog_ = true; }
            const int na0 = na_;
            while (na_ < np && ((have >> ((planpk >> (4 * na_)) & 15)) & 1u)) ++na_;
            if (na_ != na0) {
                if (na0 < 2 && na_ >= 2) STAMP(13);
                if (na_ >= 16) STAMP(15);
                lds_set(&sm.prog[6], 2 * na_);
            }
            if (newst) {
                int sidx = -1; unsigned t = newst;
#pragma unroll
                for (int q = 0; q < 4; ++q) { const int e = t ? __builtin_ctz(t) : -1; if ((lane >> 4) == q) sidx = e; t &= t - 1; }
                const unsigned vt = sidx >= 0 ? (unsigned)sidx * 32768u + (unsigned)((lane >> 3) & 1) * 16384u + (unsigned)(lane & 7) * 128u : OOB;
                sink ^= tch;
                tch = __builtin_amdgcn_raw_buffer_load_b32(rws, vt, xtouch, 0);
            }
            if (prog_) continue;
            if (__builtin_amdgcn_s_memrealtime() - t0 <= 100000ull) {
                __builtin_amdgcn_s_sleep(4);
            } else {
                const unsigned tm = __builtin_amdgcn_readfirstlane(
                    __builtin_amdgcn_raw_buffer_load_b32(rws, 0, OFF_TMO, AUX_AGT));
                if (tm == seq || __builtin_amdgcn_s_memrealtime() - t0 > DEADLINE_TICKS) {
                    if (lane == 0) {
                        __builtin_amdgcn_raw_buffer_store_b32(seq, rws, 0, OFF_TMO, AUX_AGT);
                        __hip_atomic_store(&sm.fail, 1, __ATOMIC_RELAXED, __HIP_MEMORY_SCOPE_WORKGROUP);
                    }
                    have = 0xffffu;
                }
            }
            have = __builtin_amdgcn_readfirstlane(have);
        }
        if ((sink ^ tch) == 0x9e3779b9u && seq == 0u) sm.fail = 1;
    }
    __builtin_amdgcn_s_setprio(0);
    __syncthreads();

    const unsigned long long tw0 = __builtin_amdgcn_s_memrealtime();
    auto give_up = [&]() __attribute__((always_inline)) -> bool {
        if (__builtin_amdgcn_s_memrealtime() - tw0 < 100000ull) { __builtin_amdgcn_s_sleep(1); return false; }
        const unsigned tm = __builtin_amdgcn_readfirstlane(
            __builtin_amdgcn_raw_buffer_load_b32(rws, 0, OFF_TMO, AUX_AGT));
        if (tm == seq || __builtin_amdgcn_s_memrealtime() - tw0 > DEADLINE_TICKS) {
            if (lane == 0) {
                __builtin_amdgcn_raw_buffer_store_b32(seq, rws, 0, OFF_TMO, AUX_AGT);
                __hip_atomic_store(&sm.fail, 1, __ATOMIC_RELAXED, __HIP_MEMORY_SCOPE_WORKGROUP);
            }
            return true;
        }
        return false;
    };

    const int fbase = (wave >> 1) * 8 + (wave & 1) * 4;
    f32x4 v[4];
    {
        for (;;) {
            const unsigned f = __builtin_amdgcn_raw_buffer_load_b32(rws, (unsigned)(lane & 3) * 64u,
                                                                    OFF_RF + (unsigned)(tile * 4 + g) * 256u, AUX_AGT);
            if (__all((f == seq) | ((lane & 3) == g))) break;
            if (give_up()) break;
        }
        asm volatile("" ::: "memory");
        u32x4 pr[3][2];
#pragma unroll
        for (int s = 0; s < 3; ++s) {
            const int src = s + (s >= g ? 1 : 0);
            const unsigned so = OFF_RED + (unsigned)((tile * 4 + g) * 4 + src) * 24576u + (unsigned)wave * 2048u;
#pragma unroll
            for (int i = 0; i < 2; ++i)
                pr[s][i] = __builtin_amdgcn_raw_buffer_load_b128(rws, (unsigned)lane * 16u + i * 1024u, so, AUX_AGT);
        }
#pragma unroll
        for (int i = 0; i < 4; ++i) v[i] = sm.own[(fbase + i) * 64 + lane];
#pragma unroll
        for (int s = 0; s < 3; ++s)
#pragma unroll
            for (int i = 0; i < 4; ++i) {
                const unsigned lo = pr[s][i >> 1][(i & 1) * 2], hi = pr[s][i >> 1][(i & 1) * 2 + 1];
                v[i][0] += __builtin_bit_cast(float, lo << 16);
                v[i][1] += __builtin_bit_cast(float, lo & 0xffff0000u);
                v[i][2] += __builtin_bit_cast(float, hi << 16);
                v[i][3] += __builtin_bit_cast(float, hi & 0xffff0000u);
            }
    }
    float vmax = 0.f;
#pragma unroll
    for (int i = 0; i < 4; ++i)
#pragma unroll
        for (int r = 0; r < 4; ++r) { v[i][r] = fmaxf(v[i][r], 0.f); vmax = fmaxf(vmax, v[i][r]); }
    vmax = wave_max(vmax);
    if (lane == 0) sm.wmax[wave] = vmax;
    __syncthreads();
    if (wave == 0) STAMP(7);

    {
        float m = sm.wmax[0];
#pragma unroll
        for (int q = 1; q < 8; ++q) m = fmaxf(m, sm.wmax[q]);
        if (jb != 0) {
            if (wave == 1 && lane == 0) {
                u32x2 gq; gq[0] = __builtin_bit_cast(unsigned, m); gq[1] = seq;
                __builtin_amdgcn_raw_buffer_store_b64(gq, rws, 0, OFF_AM + (unsigned)((b & 7) * 32 + jb) * 8u, AUX_AGT);
            }
        } else if (wave == 1) {
            const unsigned vg = (lane >= 1 && lane < 32) ? (unsigned)lane * 8u : OOB;
            float sm_ = m;
            for (;;) {
                const u32x2 gq = __builtin_amdgcn_raw_buffer_load_b64(rws, vg, OFF_AM + (unsigned)(b & 7) * 256u, AUX_AGT);
                sm_ = fmaxf(m, __builtin_bit_cast(float, gq[0]));
                if (__all((gq[1] == seq) | (vg == OOB))) break;
                if (give_up()) break;
            }
            sm_ = wave_max(sm_);
            u32x2 gq; gq[0] = __builtin_bit_cast(unsigned, sm_); gq[1] = seq;
#pragma unroll
            for (int q = 0; q < NRK; ++q) {
                const rsrc_t rq = mkrsrc(P.ws[q], WS_USED);
                if (lane == 0)
                    __builtin_amdgcn_raw_buffer_store_b64(gq, rq, 0, OFF_AM + 4096u + (unsigned)(rank * 8 + (b & 7)) * 8u, AUX_SYS);
            }
        }
        if (wave == 0) {
            float gm = 0.f;
            for (;;) {
                const u32x2 gq = __builtin_amdgcn_raw_buffer_load_b64(rws, (unsigned)lane * 8u, OFF_AM + 4096u, AUX_SYS);
                gm = __builtin_bit_cast(float, gq[0]);
                if (__all(gq[1] == seq)) break;
                if (give_up()) break;
            }
            gm = wave_max(gm);
            if (lane == 0) sm.gmax = gm;
        }
    }
    __syncthreads();
    if (wave == 0) STAMP(8);

    const float amax = sm.gmax;
    const unsigned tmw = __builtin_amdgcn_readfirstlane(__builtin_amdgcn_raw_buffer_load_b32(rws, 0, OFF_TMO, AUX_AGT));
    const bool failed = (sm.fail != 0) | (tmw == seq);
    const float inv = amax > 0.f ? 448.0f / amax : 0.f;
    const rsrc_t ro = mkrsrc(P.out, 512u * 4096u);
#pragma unroll
    for (int r = 0; r < 4; ++r) {
        const float q0 = fminf(v[0][r] * inv, 448.f), q1 = fminf(v[1][r] * inv, 448.f);
        const float q2 = fminf(v[2][r] * inv, 448.f), q3 = fminf(v[3][r] * inv, 448.f);
        int pk = __builtin_amdgcn_cvt_pk_fp8_f32(q0, q1, 0, false);
        pk = __builtin_amdgcn_cvt_pk_fp8_f32(q2, q3, pk, true);
        if (failed) pk = 0x7f7f7f7f;
        const unsigned row = (unsigned)(mh * 256 + g * 64 + (wave >> 1) * 16 + (lane >> 4) * 4 + r);
        __builtin_amdgcn_raw_buffer_store_b32((unsigned)pk, ro, row * 4096u + (unsigned)(lane & 15) * 4u,
                                              (unsigned)(ntile * 128 + (wave & 1) * 64), 0);
    }
    if (tid == 0) __builtin_amdgcn_raw_buffer_store_b32(seq, rws, 0, OFF_CNT + b * 128u, AUX_AGT);
}

}

extern "C" void kernel_launch(int rank, int nranks, ncclComm_t comm,
                              hipStream_t stream,
                              void* const* d_in, const int* in_sizes, int n_in,
                              void* d_out, int out_size,
                              const int* in_dtypes, int out_dtype,
                              void* const* d_out_all,
                              void* const* d_ws_all, size_t ws_bytes) {
    if (nranks != NRK || (unsigned)rank >= (unsigned)NRK || ws_bytes < WS_USED) {
        fprintf(stderr, "unexpected configuration\n");
        return;
    }
    Params P;
    P.x = (const float*)d_in[0];
    P.w = (const float*)d_in[1];
    P.out = (unsigned char*)d_out;
    for (int q = 0; q < NRK; ++q) P.ws[q] = (char*)d_ws_all[q];
    P.myws = (char*)d_ws_all[rank];
    P.rank = rank;
    a2a_gemm_kernel<<<dim3(NWG), dim3(512), 0, stream>>>(P);
}
```

```cpp
#include <hip/hip_runtime.h>
#include <rccl/rccl.h>
#include <cstdio>
#include <cstdint>

typedef float    f32x4 __attribute__((ext_vector_type(4)));
typedef float    f32x2 __attribute__((ext_vector_type(2)));
typedef unsigned u32x4 __attribute__((ext_vector_type(4)));
typedef unsigned u32x2 __attribute__((ext_vector_type(2)));
typedef int      i32x4 __attribute__((ext_vector_type(4)));
typedef __bf16   bf16x8 __attribute__((ext_vector_type(8)));
typedef __bf16   bf16x2 __attribute__((ext_vector_type(2)));

#define UNTAG 1
#define LATEX 1

namespace {

constexpr int NRK   = 8;
constexpr int NWG   = 256;
constexpr int NUN   = 32;
constexpr int NU    = 12;

constexpr unsigned OFF_X   = 0;
constexpr unsigned OFF_XF  = 4u << 20;
constexpr unsigned OFF_AM  = OFF_XF + 16384u;
constexpr unsigned OFF_CNT = OFF_AM + 16384u;
constexpr unsigned OFF_RF  = OFF_CNT + 32768u;
constexpr unsigned OFF_TMO = OFF_RF + 65536u;
constexpr unsigned OFF_RED = 8u << 20;
constexpr unsigned WS_USED = OFF_RED + (24u << 20);

#define STAMP(i) do {} while (0)
#define VSTAMP(i, v) do {} while (0)
#define CLK() 0ull
constexpr int AUX_SYS = 17;
constexpr int AUX_AGT = 16;
constexpr unsigned long long DEADLINE_TICKS = 50000000ull;
constexpr unsigned OOB = 0x40000000u;

struct Params {
    const float* x;
    const float* w;
    unsigned char* out;
    char* myws;
    char* ws[NRK];
    int rank;
};

typedef __amdgpu_buffer_rsrc_t rsrc_t;

__device__ __forceinline__ rsrc_t mkrsrc(const void* p, unsigned bytes) {
    return __builtin_amdgcn_make_buffer_rsrc(const_cast<void*>(p), (short)0, (int)bytes, 0x00020000);
}
__device__ __forceinline__ unsigned pk2(float a, float b) {
    bf16x2 r = __builtin_convertvector((f32x2){a, b}, bf16x2);
    return __builtin_bit_cast(unsigned, r);
}
__device__ __forceinline__ float wave_max(float v) {
#pragma unroll
    for (int o = 32; o >= 1; o >>= 1) v = fmaxf(v, __shfl_xor(v, o, 64));
    return v;
}
template <int N> __device__ __forceinline__ void wait_vm() {
    asm volatile("s_waitcnt vmcnt(%0)" :: "i"(N) : "memory");
}
__device__ __forceinline__ void lds_set(int* p, int v) {
    __hip_atomic_store(p, v, __ATOMIC_RELAXED, __HIP_MEMORY_SCOPE_WORKGROUP);
}
__device__ __forceinline__ i32x4 lds_ld4(const int* p) {
    typedef const volatile __attribute__((address_space(3))) i32x4* lp_t;
    return *(lp_t)(p);
}
__device__ __forceinline__ int cons_done(const int* p) {
    const i32x4 v = lds_ld4(p);
    return __builtin_amdgcn_readfirstlane(min(min(v[0], v[1]), min(v[2], v[3])));
}

__device__ __forceinline__ int stage_of(int c, int r) {
    const int g = c & 3, j = c >> 2;
    if (r == 0) return j;
    const int t = (c - r) & 7;
    return 2 + 2 * t + j - (((g < t) || (g == t && j == 1)) ? 1 : 0) - ((g + 4 < t) ? 1 : 0);
}

__device__ __forceinline__ f32x4 acc_rd(const f32x4& a) {
    float r0, r1, r2, r3;
    asm volatile("s_nop 1\n\tv_accvgpr_read_b32 %0, %4\n\tv_accvgpr_read_b32 %1, %5\n\tv_accvgpr_read_b32 %2, %6\n\tv_accvgpr_read_b32 %3, %7"
                 : "=v"(r0), "=v"(r1), "=v"(r2), "=v"(r3) : "a"(a[0]), "a"(a[1]), "a"(a[2]), "a"(a[3]));
    return (f32x4){r0, r1, r2, r3};
}
__device__ __forceinline__ u32x4 cvt8(const f32x4& a, const f32x4& b) {
    u32x4 d;
    d[0] = pk2(a[0], a[1]); d[1] = pk2(a[2], a[3]); d[2] = pk2(b[0], b[1]); d[3] = pk2(b[2], b[3]);
    return d;
}
struct Smem {
    u32x4 ring[NU][512];
    f32x4 own[32 * 64];
    int prog[8];
    int tab[16];
    int fail;
    float wmax[8], gmax;
};

__global__ void __launch_bounds__(512, 2) a2a_gemm_kernel(Params P) {
    __shared__ __attribute__((aligned(16))) Smem sm;

    const int tid  = threadIdx.x;
    const int lane = tid & 63;
    const int wave = __builtin_amdgcn_readfirstlane(tid >> 6);
    const int b    = blockIdx.x;
    const int rank = P.rank;

    const rsrc_t rws = mkrsrc(P.myws, WS_USED);
    const int jb    = b >> 3;
    const int g     = (b & 7) >> 1;
    const int ntile = (b & 1) * 16 + (jb >> 1);
    const int mh    = jb & 1;
    const int tile  = ntile * 2 + mh;

    const int d   = (b + (b >> 3)) & 7;
    const int ph  = b >> 6;
    const int pj  = (b >> 3) & 7;
    const int pr_ = (rank - d) & 7;
    const unsigned long long tk0 = __builtin_amdgcn_s_memrealtime();
    f32x4 xv[32];
    auto load_x = [&]() __attribute__((always_inline)) {
        const rsrc_t rx = mkrsrc(P.x, 4096u * 512u * 4u);
        const unsigned vox = (unsigned)(512 * d + 64 * pj + (lane & 15)) * 2048u + (unsigned)(lane >> 4) * 32u;
#pragma unroll
        for (int q = 0; q < 4; ++q) {
            const unsigned xo = (unsigned)((2 * ph + (q >> 1) + pr_) & 7) * 256u + (unsigned)(q & 1) * 128u;
#pragma unroll
            for (int mf = 0; mf < 4; ++mf) {
                xv[8 * q + 2 * mf]     = __builtin_bit_cast(f32x4, __builtin_amdgcn_raw_buffer_load_b128(rx, vox, xo + mf * 32768u, 0));
                xv[8 * q + 2 * mf + 1] = __builtin_bit_cast(f32x4, __builtin_amdgcn_raw_buffer_load_b128(rx, vox, xo + mf * 32768u + 16u, 0));
            }
        }
    };
    const bool early = (pr_ == 0) | (ph == 0);
    if (wave == 6 && early) load_x();
    unsigned seq = __builtin_amdgcn_raw_buffer_load_b32(rws, 0, OFF_CNT + b * 128u, AUX_AGT);
    if (tid < 8) sm.prog[tid] = 0;
    if (tid == 32) sm.fail = 0;
    if (tid >= 64 && tid < 80) {
        const int i = tid - 64, t = i >> 1, j = i & 1;
        const int r = (g + 4 * j - t) & 7;
        sm.tab[stage_of(g + 4 * j, r)] = 512 * ((rank + r) & 7) + 64 * (g + 4 * j);
    }
    seq = __builtin_amdgcn_readfirstlane(seq) + 1u;
    __syncthreads();
    const unsigned tag = ((seq & 0x7fffu) | 0x8000u) << 16;

    if (wave < 4) {
        f32x4 acc[4][8];
#pragma unroll
        for (int f = 0; f < 4; ++f)
#pragma unroll
            for (int i = 0; i < 8; ++i) acc[f][i] = (f32x4){0.f, 0.f, 0.f, 0.f};
        const unsigned vox = (unsigned)lane * 16u;
        const unsigned xbase = OFF_X + (unsigned)((mh * 4 + g) * 16) * 32768u + (unsigned)wave * 4096u;
        u32x4 a0[4], a1[4], a2[4], a3[4];
        bf16x8 fb[8];
        unsigned long long cw = 0;
        int rdw = 0, rdx = 0;
        auto need = [&](int nw, int nx_) __attribute__((always_inline)) {
            if (rdw < nw || rdx < nx_) {
                unsigned long long t_ = CLK();
                for (;;) {
                    const i32x4 v = lds_ld4(&sm.prog[4]);
                    rdw = __builtin_amdgcn_readfirstlane(min(v[0], v[1]));
                    rdx = __builtin_amdgcn_readfirstlane(v[2]);
                    if (rdw >= nw && rdx >= nx_) break;
                    __builtin_amdgcn_s_sleep(1);
                }
                cw += CLK() - t_;
            }
            asm volatile("" ::: "memory");
        };
        auto issueA = [&](u32x4 (&av_)[4], int n) __attribute__((always_inline)) {
            const unsigned sx = xbase + (unsigned)n * 16384u;
#pragma unroll
            for (int f = 0; f < 4; ++f)
                av_[f] = __builtin_amdgcn_raw_buffer_load_b128(rws, vox + f * 1024u, sx, AUX_SYS);
        };
        auto block = [&](const u32x4 (&ca)[4], u32x4 (&na)[4], int u, int nslot) __attribute__((always_inline)) {
            need(u + 2 < NUN ? u + 2 : NUN, u + 4 < NUN ? u + 4 : NUN);
            __builtin_amdgcn_sched_barrier(0);
            const unsigned sx = u + 3 < NUN ? xbase + (unsigned)(u + 3) * 16384u : OOB;
            const u32x4* rb = &sm.ring[nslot][lane];
            i32x4 fl_v;
#pragma unroll
            for (int i = 0; i < 8; ++i) {
#pragma unroll
                for (int f = 0; f < 4; ++f)
                    asm volatile("v_mfma_f32_16x16x32_bf16 %0, %1, %2, %0" : "+a"(acc[f][i]) : "v"(ca[f]), "v"(fb[i]));
                __builtin_amdgcn_sched_barrier(0);
                fb[i] = __builtin_bit_cast(bf16x8, rb[i * 64]);
                if (i < 4) na[i] = __builtin_amdgcn_raw_buffer_load_b128(rws, vox + i * 1024u, sx, AUX_SYS);
                if (i == 5) fl_v = lds_ld4(&sm.prog[4]);
                __builtin_amdgcn_sched_barrier(0);
            }
            lds_set(&sm.prog[wave], u + 1);
            rdw = __builtin_amdgcn_readfirstlane(min(fl_v[0], fl_v[1]));
            rdx = __builtin_amdgcn_readfirstlane(fl_v[2]);
        };
        need(0, 1); issueA(a0, 0);
        need(0, 2); issueA(a1, 1);
        need(0, 3); issueA(a2, 2);
        need(1, 0);
        if (wave == 0) STAMP(2);
#pragma unroll
        for (int i = 0; i < 8; ++i) fb[i] = __builtin_bit_cast(bf16x8, sm.ring[0][i * 64 + lane]);
        unsigned long long c0 = CLK();
        int slot = 0;
        for (int u = 0; u < NUN; u += 4) {
            if (wave == 0) { if (u == 8) STAMP(11); if (u == 16) STAMP(12); if (u == 24) STAMP(14); }
            block(a0, a3, u, slot + 1);
            block(a1, a0, u + 1, slot + 2);
            block(a2, a1, u + 2, slot + 3);
            slot += 4; if (slot == NU) slot = 0;
            block(a3, a2, u + 3, slot);
        }
        asm volatile("s_nop 15\n\ts_nop 15"
                     : "+a"(acc[0][6]), "+a"(acc[1][6]), "+a"(acc[2][6]), "+a"(acc[3][6]),
                       "+a"(acc[0][7]), "+a"(acc[1][7]), "+a"(acc[2][7]), "+a"(acc[3][7]) :: "memory");
        if (wave == 0) { VSTAMP(0, CLK() - c0); VSTAMP(1, cw); STAMP(3); }
        if (wave == g) {
#pragma unroll
            for (int f = 0; f < 4; ++f)
#pragma unroll
                for (int i = 0; i < 8; ++i) { sm.own[(f * 8 + i) * 64 + lane] = acc_rd(acc[f][i]); __builtin_amdgcn_sched_barrier(0); }
        } else {
            const unsigned so = OFF_RED + (unsigned)((tile * 4 + wave) * 4 + g) * 24576u;
#pragma unroll
            for (int f = 0; f < 4; ++f)
#pragma unroll
                for (int ip = 0; ip < 4; ++ip) {
                    __builtin_amdgcn_raw_buffer_store_b128(cvt8(acc_rd(acc[f][2 * ip]), acc_rd(acc[f][2 * ip + 1])), rws, vox + ip * 1024u, so + f * 4096u, AUX_AGT);
                    __builtin_amdgcn_sched_barrier(0);
                }
            wait_vm<0>();
            if (lane == 0)
                __builtin_amdgcn_raw_buffer_store_b32(seq, rws, 0, OFF_RF + (unsigned)((tile * 4 + wave) * 4 + g) * 64u, AUX_AGT);
        }
    } else if (wave < 6) {
        __builtin_amdgcn_s_setprio(2);
        const int nh = wave - 4;
        const rsrc_t rw = mkrsrc(P.w, 4096u * 4096u * 4u);
        const unsigned vow = (unsigned)(lane >> 4) * (8u * 16384u) + (unsigned)(lane & 15) * 16u;
        const unsigned wcol = (unsigned)(ntile * 128 + nh * 64) * 4u;
        unsigned long long lw = 0, l0 = CLK();
        f32x4 wb[3][8];
        const int tabv = sm.tab[lane & 15];
        auto issueW = [&](f32x4 (&wv)[8], int u) __attribute__((always_inline)) {
            unsigned sw = OOB;
            if (u < NUN) sw = (unsigned)(__builtin_amdgcn_readlane(tabv, u >> 1) + 32 * (u & 1)) * 16384u + wcol;
#pragma unroll
            for (int j = 0; j < 8; ++j)
                wv[j] = __builtin_bit_cast(f32x4, __builtin_amdgcn_raw_buffer_load_b128(rw, vow, sw + (unsigned)j * 16384u, 0));
        };
        int cdone = 0;
        auto fill = [&](const f32x4 (&wv)[8], int slot, int u) __attribute__((always_inline)) {
            if (cdone < u - (NU - 1)) {
                unsigned long long t_ = CLK();
                while ((cdone = cons_done(&sm.prog[0])) < u - (NU - 1)) __builtin_amdgcn_s_sleep(1);
                lw += CLK() - t_;
            }
            asm volatile("" ::: "memory");
#pragma unroll
            for (int i = 0; i < 4; ++i) {
                u32x4 t;
                t[0] = pk2(wv[0][i], wv[1][i]);
                t[1] = pk2(wv[2][i], wv[3][i]);
                t[2] = pk2(wv[4][i], wv[5][i]);
                t[3] = pk2(wv[6][i], wv[7][i]);
                sm.ring[slot][(nh * 4 + i) * 64 + lane] = t;
            }
            asm volatile("" ::: "memory");
            lds_set(&sm.prog[4 + nh], u + 1);
            if (nh == 0 && u == 7) STAMP(4);
            if (nh == 0 && u == 15) STAMP(5);
            if (nh == 0 && u == 23) STAMP(6);
        };
#pragma unroll
        for (int dd = 0; dd < 3; ++dd) issueW(wb[dd], dd);
        int fslot = 0;
        for (int u = 0; u < NUN; u += 3) {
#pragma unroll
            for (int dd = 0; dd < 3; ++dd) {
                if (u + dd < NUN) fill(wb[dd], fslot + dd, u + dd);
                issueW(wb[dd], u + dd + 3);
            }
            fslot += 3; if (fslot == NU) fslot = 0;
        }
        if (nh == 0) { VSTAMP(3, CLK() - l0); VSTAMP(4, lw); STAMP(10); }
    } else if (wave == 6) {
        __builtin_amdgcn_s_setprio(2);
        unsigned so[2], fo[2];
#pragma unroll
        for (int t = 0; t < 2; ++t) {
            const int c = (2 * ph + t + pr_) & 7;
            const unsigned blk = (unsigned)(((pj >> 2) * 4 + (c & 3)) * 16 + stage_of(c, pr_));
            so[t] = OFF_X + blk * 32768u + (unsigned)(pj & 3) * 4096u;
            fo[t] = OFF_XF + (blk * 32u + (unsigned)(pj & 3) * 4u) * 4u;
        }
        char* dst = P.myws;
#pragma unroll
        for (int q = 0; q < NRK; ++q) if (q == d) dst = P.ws[q];
        const rsrc_t rdst = mkrsrc(dst, WS_USED);
#define PT1 400
#define PT2 800
#define PT3 1200
        if (pr_ != 0) {
            const unsigned long long tgo = ph == 0 ? 0 : ph == 1 ? PT1 : ph == 2 ? PT2 : PT3;
            if (!early) {
                while (__builtin_amdgcn_s_memrealtime() - tk0 + 150 < tgo) __builtin_amdgcn_s_sleep(2);
                load_x();
            }
            while (__builtin_amdgcn_s_memrealtime() - tk0 < tgo) __builtin_amdgcn_s_sleep(2);
        }
#pragma unroll
        for (int J = 0; J < 16; ++J)
            __builtin_amdgcn_raw_buffer_store_b128(cvt8(xv[2 * J], xv[2 * J + 1]), rdst, (unsigned)lane * 16u,
                                                   so[J >> 3] + ((J >> 2) & 1) * 16384u + (J & 3) * 1024u, AUX_SYS);
#pragma unroll
        for (int q = 0; q < 4; ++q) {
            if (q == 0) wait_vm<12>(); else if (q == 1) wait_vm<9>(); else if (q == 2) wait_vm<6>(); else wait_vm<3>();
            if (lane == 0) {
                u32x4 fl = {seq, seq, seq, seq};
                __builtin_amdgcn_raw_buffer_store_b128(fl, rdst, 0, fo[q >> 1] + (q & 1) * 64u, AUX_SYS);
            }
        }
        STAMP(1);
    } else {
        __builtin_amdgcn_s_setprio(2);
        const unsigned long long t0 = __builtin_amdgcn_s_memrealtime();
        const unsigned fo = OFF_XF + (unsigned)(mh * 4 + g) * 2048u;
        const unsigned xtouch = OFF_X + (unsigned)((mh * 4 + g) * 16) * 32768u + (unsigned)(ntile & 15) * 1024u;
        unsigned sink = 0;
        int av = 0;
        while (av < NUN) {
            const unsigned vf = (lane < 16 && av + (lane >> 1) < NUN) ? (unsigned)lane * 32u : OOB;
            const unsigned sf = fo + (unsigned)av * 64u;
            u32x4 f0 = __builtin_amdgcn_raw_buffer_load_b128(rws, vf, sf, AUX_SYS);
            u32x4 f1 = __builtin_amdgcn_raw_buffer_load_b128(rws, vf + 16u, sf, AUX_SYS);
            const bool ok = (f0[0] == seq) & (f0[1] == seq) & (f0[2] == seq) & (f0[3] == seq) &
                            (f1[0] == seq) & (f1[1] == seq) & (f1[2] == seq) & (f1[3] == seq);
            const unsigned m = (unsigned)__ballot(ok) & 0xffffu;
            const unsigned m2 = m & (m >> 1);
            const unsigned xm = (~m2 & 0x5555u) | 0x10000u;
            const int na = __builtin_amdgcn_readfirstlane(av + (__builtin_ctz(xm) >> 1));
            if (na > av) {
                if (av < 4 && na >= 4) STAMP(13);
                if (na >= NUN) STAMP(15);
                {
                    const unsigned vt = (av + (lane >> 3) < na) ? (unsigned)(lane >> 3) * 16384u + (unsigned)(lane & 7) * 128u : OOB;
                    sink ^= __builtin_amdgcn_raw_buffer_load_b32(rws, vt, xtouch + (unsigned)av * 16384u, 0);
                }
                av = na;
                lds_set(&sm.prog[6], av);
            } else if (__builtin_amdgcn_s_memrealtime() - t0 <= 100000ull) {
                __builtin_amdgcn_s_sleep(2);
            } else {
                const unsigned tm = __builtin_amdgcn_readfirstlane(
                    __builtin_amdgcn_raw_buffer_load_b32(rws, 0, OFF_TMO, AUX_AGT));
                if (tm == seq || __builtin_amdgcn_s_memrealtime() - t0 > DEADLINE_TICKS) {
                    if (lane == 0) {
                        __builtin_amdgcn_raw_buffer_store_b32(seq, rws, 0, OFF_TMO, AUX_AGT);
                        __hip_atomic_store(&sm.fail, 1, __ATOMIC_RELAXED, __HIP_MEMORY_SCOPE_WORKGROUP);
                    }
                    av = NUN;
                    lds_set(&sm.prog[6], av);
                }
            }
            av = __builtin_amdgcn_readfirstlane(av);
        }
        if (sink == 0x9e3779b9u && seq == 0u) sm.fail = 1;
    }
    __builtin_amdgcn_s_setprio(0);
    __syncthreads();

    const unsigned long long tw0 = __builtin_amdgcn_s_memrealtime();
    auto give_up = [&]() __attribute__((always_inline)) -> bool {
        if (__builtin_amdgcn_s_memrealtime() - tw0 < 100000ull) { __builtin_amdgcn_s_sleep(1); return false; }
        const unsigned tm = __builtin_amdgcn_readfirstlane(
            __builtin_amdgcn_raw_buffer_load_b32(rws, 0, OFF_TMO, AUX_AGT));
        if (tm == seq || __builtin_amdgcn_s_memrealtime() - tw0 > DEADLINE_TICKS) {
            if (lane == 0) {
                __builtin_amdgcn_raw_buffer_store_b32(seq, rws, 0, OFF_TMO, AUX_AGT);
                __hip_atomic_store(&sm.fail, 1, __ATOMIC_RELAXED, __HIP_MEMORY_SCOPE_WORKGROUP);
            }
            return true;
        }
        return false;
    };

    const int fbase = (wave >> 1) * 8 + (wave & 1) * 4;
    f32x4 v[4];
    {
        for (;;) {
            const unsigned f = __builtin_amdgcn_raw_buffer_load_b32(rws, (unsigned)(lane & 3) * 64u,
                                                                    OFF_RF + (unsigned)(tile * 4 + g) * 256u, AUX_AGT);
            if (__all((f == seq) | ((lane & 3) == g))) break;
            if (give_up()) break;
        }
        asm volatile("" ::: "memory");
        u32x4 pr[3][2];
#pragma unroll
        for (int s = 0; s < 3; ++s) {
            const int src = s + (s >= g ? 1 : 0);
            const unsigned so = OFF_RED + (unsigned)((tile * 4 + g) * 4 + src) * 24576u + (unsigned)wave * 2048u;
#pragma unroll
            for (int i = 0; i < 2; ++i)
                pr[s][i] = __builtin_amdgcn_raw_buffer_load_b128(rws, (unsigned)lane * 16u + i * 1024u, so, AUX_AGT);
        }
#pragma unroll
        for (int i = 0; i < 4; ++i) v[i] = sm.own[(fbase + i) * 64 + lane];
#pragma unroll
        for (int s = 0; s < 3; ++s)
#pragma unroll
            for (int i = 0; i < 4; ++i) {
                const unsigned lo = pr[s][i >> 1][(i & 1) * 2], hi = pr[s][i >> 1][(i & 1) * 2 + 1];
                v[i][0] += __builtin_bit_cast(float, lo << 16);
                v[i][1] += __builtin_bit_cast(float, lo & 0xffff0000u);
                v[i][2] += __builtin_bit_cast(float, hi << 16);
                v[i][3] += __builtin_bit_cast(float, hi & 0xffff0000u);
            }
    }
    float vmax = 0.f;
#pragma unroll
    for (int i = 0; i < 4; ++i)
#pragma unroll
        for (int r = 0; r < 4; ++r) { v[i][r] = fmaxf(v[i][r], 0.f); vmax = fmaxf(vmax, v[i][r]); }
    vmax = wave_max(vmax);
    if (lane == 0) sm.wmax[wave] = vmax;
    __syncthreads();
    if (wave == 0) STAMP(7);

    {
        float m = sm.wmax[0];
#pragma unroll
        for (int q = 1; q < 8; ++q) m = fmaxf(m, sm.wmax[q]);
        if (jb != 0) {
            if (wave == 1 && lane == 0) {
                u32x2 gq; gq[0] = __builtin_bit_cast(unsigned, m); gq[1] = seq;
                __builtin_amdgcn_raw_buffer_store_b64(gq, rws, 0, OFF_AM + (unsigned)((b & 7) * 32 + jb) * 8u, AUX_AGT);
            }
        } else if (wave == 1) {
            const unsigned vg = (lane >= 1 && lane < 32) ? (unsigned)lane * 8u : OOB;
            float sm_ = m;
            for (;;) {
                const u32x2 gq = __builtin_amdgcn_raw_buffer_load_b64(rws, vg, OFF_AM + (unsigned)(b & 7) * 256u, AUX_AGT);
                sm_ = fmaxf(m, __builtin_bit_cast(float, gq[0]));
                if (__all((gq[1] == seq) | (vg == OOB))) break;
                if (give_up()) break;
            }
            sm_ = wave_max(sm_);
            u32x2 gq; gq[0] = __builtin_bit_cast(unsigned, sm_); gq[1] = seq;
#pragma unroll
            for (int q = 0; q < NRK; ++q) {
                const rsrc_t rq = mkrsrc(P.ws[q], WS_USED);
                if (lane == 0)
                    __builtin_amdgcn_raw_buffer_store_b64(gq, rq, 0, OFF_AM + 4096u + (unsigned)(rank * 8 + (b & 7)) * 8u, AUX_SYS);
            }
        }
        if (wave == 0) {
            float gm = 0.f;
            for (;;) {
                const u32x2 gq = __builtin_amdgcn_raw_buffer_load_b64(rws, (unsigned)lane * 8u, OFF_AM + 4096u, AUX_SYS);
                gm = __builtin_bit_cast(float, gq[0]);
                if (__all(gq[1] == seq)) break;
                if (give_up()) break;
            }
            gm = wave_max(gm);
            if (lane == 0) sm.gmax = gm;
        }
    }
    __syncthreads();
    if (wave == 0) STAMP(8);

    const float amax = sm.gmax;
    const unsigned tmw = __builtin_amdgcn_readfirstlane(__builtin_amdgcn_raw_buffer_load_b32(rws, 0, OFF_TMO, AUX_AGT));
    const bool failed = (sm.fail != 0) | (tmw == seq);
    const float inv = amax > 0.f ? 448.0f / amax : 0.f;
    const rsrc_t ro = mkrsrc(P.out, 512u * 4096u);
#pragma unroll
    for (int r = 0; r < 4; ++r) {
        const float q0 = fminf(v[0][r] * inv, 448.f), q1 = fminf(v[1][r] * inv, 448.f);
        const float q2 = fminf(v[2][r] * inv, 448.f), q3 = fminf(v[3][r] * inv, 448.f);
        int pk = __builtin_amdgcn_cvt_pk_fp8_f32(q0, q1, 0, false);
        pk = __builtin_amdgcn_cvt_pk_fp8_f32(q2, q3, pk, true);
        if (failed) pk = 0x7f7f7f7f;
        const unsigned row = (unsigned)(mh * 256 + g * 64 + (wave >> 1) * 16 + (lane >> 4) * 4 + r);
        __builtin_amdgcn_raw_buffer_store_b32((unsigned)pk, ro, row * 4096u + (unsigned)(lane & 15) * 4u,
                                              (unsigned)(ntile * 128 + (wave & 1) * 64), 0);
    }
    if (tid == 0) __builtin_amdgcn_raw_buffer_store_b32(seq, rws, 0, OFF_CNT + b * 128u, AUX_AGT);
}

}

extern "C" void kernel_launch(int rank, int nranks, ncclComm_t comm,
                              hipStream_t stream,
                              void* const* d_in, const int* in_sizes, int n_in,
                              void* d_out, int out_size,
                              const int* in_dtypes, int out_dtype,
                              void* const* d_out_all,
                              void* const* d_ws_all, size_t ws_bytes) {
    if (nranks != NRK || (unsigned)rank >= (unsigned)NRK || ws_bytes < WS_USED) {
        fprintf(stderr, "unexpected configuration\n");
        return;
    }
    Params P;
    P.x = (const float*)d_in[0];
    P.w = (const float*)d_in[1];
    P.out = (unsigned char*)d_out;
    for (int q = 0; q < NRK; ++q) P.ws[q] = (char*)d_ws_all[q];
    P.myws = (char*)d_ws_all[rank];
    P.rank = rank;
    a2a_gemm_kernel<<<dim3(NWG), dim3(512), 0, stream>>>(P);
}
```
